```python
import numpy as np

kernel_name = "gpu_gemm_rs_m4096_k4096_n32768_f32_silu"


ABSMAX_RTOL = 0.001

COMPUTE_DTYPE = "float32"
EPILOGUE = "silu"
IN_QUANT = None
OUT_QUANT = None
IN_DTYPES = {"x": "float32", "w_mat": "float32"}
WORLD_SIZE = 8

def setup_inputs():

    x = np.random.randn(4096, 4096).astype(np.float32)
    w_mat = (np.random.randn(4096, 32768) / np.sqrt(4096)).astype(np.float32)
    return {"x": x, "w_mat": w_mat}

def _epilogue(y):
    return (y / (1.0 + np.exp(-np.clip(y, -60.0, 60.0)))).astype(np.float32)

def reference(x, w_mat):
    y = (x @ w_mat).astype(np.float32)
    return _epilogue(y)
```

```cpp
#include <hip/hip_runtime.h>
#include <rccl/rccl.h>
#include <cstdio>
#include <cstdint>

#define HIP_CHECK(expr) do { hipError_t _e = (expr); if (_e != hipSuccess) { \
    fprintf(stderr, "HIP error %s at %s:%d\n", hipGetErrorString(_e), __FILE__, __LINE__); \
    return; } } while (0)

#define LAS __attribute__((address_space(3)))
typedef _Float16 h8 __attribute__((ext_vector_type(8)));
typedef float f32x4 __attribute__((ext_vector_type(4)));
typedef unsigned u32x4 __attribute__((ext_vector_type(4)));

namespace {

constexpr int NR = 8;
constexpr int MG = 4096, KG = 4096, NG = 32768;
constexpr int KS = KG / NR;
constexpr int NS = NG / NR;
constexpr int BM = 256, BK = 64, HALF = 128;
constexpr int NMT = MG / BM;
constexpr int NNT = NS / BM;
constexpr int NKT = KG / BK;
constexpr int KTS = KS / BK;
constexpr int HTB = HALF * BK * 2;
constexpr int TB = 2 * HTB;
constexpr int LDS_BYTES = 8 * HTB;
constexpr int NTHR = 512;
constexpr int GRID = 256;

constexpr size_t L_TICK0 = 0 * 128, L_TICK1 = 1 * 128, L_TICK2 = 2 * 128, L_TMO = 3 * 128, L_DONE = 4 * 128, L_INRDY = 5 * 128;
constexpr size_t L_XRDY = 16 * 128;
constexpr size_t L_WRDY = 32 * 128;
constexpr size_t O_EX = 64 * 1024;
constexpr size_t O_EW = O_EX + 2 * 16 * 64 * 4;
constexpr size_t O_XS = 1 << 20;
constexpr size_t PANEL = (size_t)NKT * TB;
constexpr size_t OPB = (size_t)16 * PANEL;
constexpr size_t O_WS = O_XS + 2 * OPB;
constexpr size_t WS_NEED = O_WS + 2 * OPB;

constexpr unsigned long long DEADLINE = 150000000ull;

struct Params {
    const float* x;
    const float* w;
    float* out[NR];
    char* ws[NR];
    int rank;
    int nprod;
    unsigned char tile_of_block[GRID];
};

__host__ __device__ __forceinline__ int lds_byte(int r, int c) {
    const int st = (r >> 4) * 2 + (c >> 5), rr = r & 15, cc = c & 31, ob = rr * 64 + cc * 2;
    return st * 1024 + (ob ^ (((ob >> 9) & 1) << 5));
}
__host__ __device__ __forceinline__ void stage_rc(int b, int& R, int& C) {
    const int st = b / 1024, sb = b % 1024, swz = sb ^ (((sb >> 9) & 1) << 5);
    R = (st >> 1) * 16 + swz / 64; C = (st & 1) * 32 + (swz % 64) / 2;
}

__device__ __forceinline__ unsigned* ctl(char* ws, size_t off) { return (unsigned*)(ws + off); }

__device__ __forceinline__ unsigned call_number(char* myws, size_t site, unsigned nwg, LAS unsigned* sh) {
    if (threadIdx.x == 0) {
        unsigned t = __hip_atomic_fetch_add(ctl(myws, site), 1u, __ATOMIC_RELAXED, __HIP_MEMORY_SCOPE_AGENT);
        sh[0] = t / nwg + 1;
    }
    __syncthreads();
    unsigned c = sh[0];
    __syncthreads();
    return __builtin_amdgcn_readfirstlane(c);
}

__device__ __forceinline__ void wait_flag(unsigned* flag, unsigned target, unsigned* tmo, unsigned call) {
    unsigned long long t0 = __builtin_amdgcn_s_memrealtime();
    while (__hip_atomic_load(flag, __ATOMIC_RELAXED, __HIP_MEMORY_SCOPE_SYSTEM) < target) {
        if (__hip_atomic_load(tmo, __ATOMIC_RELAXED, __HIP_MEMORY_SCOPE_AGENT) == call
            || __builtin_amdgcn_s_memrealtime() - t0 > DEADLINE) {
            __hip_atomic_store(tmo, call, __ATOMIC_RELAXED, __HIP_MEMORY_SCOPE_AGENT);
            break;
        }
        __builtin_amdgcn_s_sleep(2);
    }
}

__device__ __forceinline__ int scale_exp(float amax) {
    unsigned b = __float_as_uint(amax);
    int ex = (int)((b >> 23) & 255);
    if (amax == 0.f || ex == 255) return 0;
    int il = ex - 127;
    int e = ((13 - il) >> 3) << 3;
    e = e > 64 ? 64 : e; e = e < -64 ? -64 : e;
    return e;
}

__device__ __forceinline__ float wg_amax(float v, LAS float* red) {
    unsigned u = __float_as_uint(v);
#pragma unroll
    for (int o = 32; o > 0; o >>= 1) { unsigned t = (unsigned)__shfl_xor((int)u, o); u = u > t ? u : t; }
    const int wid = threadIdx.x >> 6;
    __syncthreads();
    if ((threadIdx.x & 63) == 0) ((LAS unsigned*)red)[wid] = u;
    __syncthreads();
    unsigned m = 0;
#pragma unroll
    for (int i = 0; i < 8; ++i) { unsigned t = ((LAS unsigned*)red)[i]; m = m > t ? m : t; }
    return __uint_as_float(m);
}

__device__ __forceinline__ h8 cvt8(f32x4 a, f32x4 b, int e) {
    h8 r;
    r[0] = (_Float16)ldexpf(a[0], e); r[1] = (_Float16)ldexpf(a[1], e); r[2] = (_Float16)ldexpf(a[2], e); r[3] = (_Float16)ldexpf(a[3], e);
    r[4] = (_Float16)ldexpf(b[0], e); r[5] = (_Float16)ldexpf(b[1], e); r[6] = (_Float16)ldexpf(b[2], e); r[7] = (_Float16)ldexpf(b[3], e);
    return r;
}

__device__ __forceinline__ float abs4max(f32x4 a) {
    unsigned m0 = __float_as_uint(a[0]) & 0x7fffffffu, m1 = __float_as_uint(a[1]) & 0x7fffffffu;
    unsigned m2 = __float_as_uint(a[2]) & 0x7fffffffu, m3 = __float_as_uint(a[3]) & 0x7fffffffu;
    m0 = m0 > m1 ? m0 : m1; m2 = m2 > m3 ? m2 : m3; m0 = m0 > m2 ? m0 : m2;
    return __uint_as_float(m0);
}
__device__ __forceinline__ float umaxf(float a, float b) { return __float_as_uint(a) > __float_as_uint(b) ? a : b; }

__device__ __forceinline__ void produce_x(const Params& p, int par, int mt, int ktl, LAS unsigned char* lds) {
    const int tid = threadIdx.x;
    f32x4 va[4], vb[4];
    float am = 0.f;
#pragma unroll
    for (int q = 0; q < 4; ++q) {
        const int o = (q * NTHR + tid) * 16, half = o >> 14;
        int R, C; stage_rc(o & 16383, R, C);
        const float* src = p.x + (size_t)(mt * BM + half * HALF + R) * KS + ktl * BK + C;
        va[q] = *(const f32x4*)src; vb[q] = *(const f32x4*)(src + 4);
        am = umaxf(am, umaxf(abs4max(va[q]), abs4max(vb[q])));
    }
    am = wg_amax(am, (LAS float*)(lds + LDS_BYTES - 64));
    const int e = scale_exp(am);
    const int kt = p.rank * KTS + ktl;
    const size_t off = O_XS + (size_t)par * OPB + (size_t)mt * PANEL + (size_t)kt * TB;
    h8 hv[4];
#pragma unroll
    for (int q = 0; q < 4; ++q) hv[q] = cvt8(va[q], vb[q], e);
#pragma unroll
    for (int d = 0; d < NR; ++d) {
        const int dst = (p.rank + d) & (NR - 1);
        char* base = p.ws[dst] + off;
#pragma unroll
        for (int q = 0; q < 4; ++q) *(h8*)(base + (q * NTHR + tid) * 16) = hv[q];
        if (tid == 0) ((int*)(p.ws[dst] + O_EX))[(par * 16 + mt) * 64 + kt] = e;
    }
}

__device__ __forceinline__ void produce_w(const Params& p, int par, int dst, int nt, int ktl, LAS unsigned char* lds) {
    const int tid = threadIdx.x, wid = tid >> 6, lane = tid & 63;
    const int kh = wid & 1, nb = wid >> 1, kc = lane >> 4, ng = lane & 15;
    const float* src = p.w + (size_t)(ktl * BK + kh * 32 + kc * 8) * NG + dst * NS + nt * BM + nb * 64 + 4 * ng;
    f32x4 v[8];
    float am = 0.f;
#pragma unroll
    for (int j = 0; j < 8; ++j) { v[j] = __builtin_nontemporal_load((const f32x4*)(src + (size_t)j * NG)); am = umaxf(am, abs4max(v[j])); }
    am = wg_amax(am, (LAS float*)(lds + LDS_BYTES - 64));
    const int e = scale_exp(am);
#pragma unroll
    for (int i = 0; i < 4; ++i) {
        f32x4 lo = {v[0][i], v[1][i], v[2][i], v[3][i]}, hi = {v[4][i], v[5][i], v[6][i], v[7][i]};
        const int nl = nb * 64 + 4 * ng + i;
        *(LAS h8*)(lds + (nl >> 7) * HTB + lds_byte(nl & 127, kh * 32 + kc * 8)) = cvt8(lo, hi, e);
    }
    __syncthreads();
    const int kt = p.rank * KTS + ktl;
    char* base = p.ws[dst] + O_WS + (size_t)par * OPB + (size_t)nt * PANEL + (size_t)kt * TB;
#pragma unroll
    for (int q = 0; q < 4; ++q) {
        h8 t = *(LAS h8*)(lds + (q * NTHR + tid) * 16);
        *(h8*)(base + (q * NTHR + tid) * 16) = t;
    }
    if (tid == 0) ((int*)(p.ws[dst] + O_EW))[(par * 16 + nt) * 64 + kt] = e;
}

__device__ __forceinline__ void wg_release() {
    asm volatile("s_waitcnt vmcnt(0)" ::: "memory");
    __syncthreads();
    if (threadIdx.x == 0) {
        __builtin_amdgcn_fence(__ATOMIC_RELEASE, "");
        asm volatile("s_waitcnt vmcnt(0)" ::: "memory");
    }
}
__device__ __forceinline__ void sys_add(unsigned* ctr, unsigned v) {
    __hip_atomic_fetch_add(ctr, v, __ATOMIC_RELAXED, __HIP_MEMORY_SCOPE_SYSTEM);
}

__device__ __forceinline__ float silu_f(float y) {
    float c = fminf(fmaxf(y, -60.f), 60.f);
    return y / (1.f + __expf(-c));
}

__device__ __forceinline__ void gemm_tile(const Params& p, int par, int mt, int nt, unsigned call, LAS unsigned char* lds, bool bad_in) {
    char* myws = p.ws[p.rank];
    const int tid = threadIdx.x, wid = __builtin_amdgcn_readfirstlane(tid >> 6), lane = tid & 63;
    const int wr = wid >> 2, wc = wid & 3, fr = lane & 15, fq = lane >> 4;
    const char* cA = myws + O_XS + (size_t)par * OPB + (size_t)mt * PANEL;
    const char* cB = myws + O_WS + (size_t)par * OPB + (size_t)nt * PANEL;
    int ev = ((const int*)(myws + O_EX))[(par * 16 + mt) * 64 + lane] + ((const int*)(myws + O_EW))[(par * 16 + nt) * 64 + lane];
    const unsigned voff = (unsigned)tid * 16u;
    const unsigned ldsw = (unsigned)wid * 1024u;
    const int aoff = lds_byte(wr * 64 + fr, fq * 8), boff = lds_byte(wc * 32 + fr, fq * 8);
#define SA(b, h) (((b) * 2 + (h)) * HTB)
#define SB(b, h) ((4 + (b) * 2 + (h)) * HTB)
#define STAGE(bufoff, gbase) do { _Pragma("unroll") for (int _i = 0; _i < 2; ++_i) \
        __builtin_amdgcn_global_load_lds((const unsigned*)((gbase) + voff + _i * 8192), (LAS unsigned*)(lds + (bufoff) + ldsw + _i * 8192), 16, 0, 0); } while (0)
#define LDA(dst, b, h) do { _Pragma("unroll") for (int m = 0; m < 4; ++m) _Pragma("unroll") for (int k = 0; k < 2; ++k) dst[m][k] = *(const LAS h8*)(lds + SA(b, h) + aoff + m * 2048 + k * 1024); } while (0)
#define LDB(dst, b, h) do { _Pragma("unroll") for (int n = 0; n < 2; ++n) _Pragma("unroll") for (int k = 0; k < 2; ++k) dst[n][k] = *(const LAS h8*)(lds + SB(b, h) + boff + n * 2048 + k * 1024); } while (0)
#define MMA(ai, bj, At, Bt) do { __builtin_amdgcn_s_setprio(1); _Pragma("unroll") for (int m = 0; m < 4; ++m) _Pragma("unroll") for (int n = 0; n < 2; ++n) _Pragma("unroll") for (int k = 0; k < 2; ++k) \
        acc[ai][bj][m][n] = __builtin_amdgcn_mfma_f32_16x16x32_f16(Bt[n][k], At[m][k], acc[ai][bj][m][n], 0, 0, 0); __builtin_amdgcn_s_setprio(0); } while (0)
#define WAIT_V(n) asm volatile("s_waitcnt vmcnt(" #n ")" ::: "memory")
#define WAIT_L(n) asm volatile("s_waitcnt lgkmcnt(" #n ")" ::: "memory")
#define BAR __builtin_amdgcn_s_barrier()
#define SCHED __builtin_amdgcn_sched_barrier(0)
#define RESCALE(tt) do { const int _e = __builtin_amdgcn_readlane(ev, (tt)); if (_e != ecur) { const int _d = _e - ecur; ecur = _e; \
        _Pragma("unroll") for (int a = 0; a < 2; ++a) _Pragma("unroll") for (int b = 0; b < 2; ++b) _Pragma("unroll") for (int m = 0; m < 4; ++m) _Pragma("unroll") for (int n = 0; n < 2; ++n) \
        _Pragma("unroll") for (int j = 0; j < 4; ++j) acc[a][b][m][n][j] = ldexpf(acc[a][b][m][n][j], _d); } } while (0)
    f32x4 acc[2][2][4][2];
#pragma unroll
    for (int a = 0; a < 2; ++a)
#pragma unroll
        for (int b = 0; b < 2; ++b)
#pragma unroll
            for (int m = 0; m < 4; ++m)
#pragma unroll
                for (int n = 0; n < 2; ++n) acc[a][b][m][n] = (f32x4){0.f, 0.f, 0.f, 0.f};
    h8 At[4][2], B0[2][2], B1[2][2];
    int ecur = __builtin_amdgcn_readlane(ev, 0);
    STAGE(SB(0, 0), cB); STAGE(SA(0, 0), cA); STAGE(SB(0, 1), cB + HTB); STAGE(SA(0, 1), cA + HTB);
    if (wr == 1) BAR;
    WAIT_V(4); BAR;
    STAGE(SB(1, 0), cB + TB); STAGE(SA(1, 0), cA + TB); STAGE(SB(1, 1), cB + TB + HTB);
    WAIT_V(6); BAR;
    for (int t = 0; t < NKT; t += 2) {
        const bool last = (t == NKT - 2);
        const char* a1 = cA + (size_t)(t + 1) * TB;
        const char* a2 = last ? cA : cA + (size_t)(t + 2) * TB; const char* b2 = last ? cB : cB + (size_t)(t + 2) * TB;
        const char* a3 = a2 + TB; const char* b3 = b2 + TB;
        RESCALE(t);
        LDB(B0, 0, 0); SCHED; LDA(At, 0, 0); STAGE(SA(1, 1), a1 + HTB);
        WAIT_L(8); BAR; WAIT_L(0); MMA(0, 0, At, B0); BAR; SCHED;
        LDB(B1, 0, 1); STAGE(SB(0, 0), b2);
        BAR; WAIT_L(0); MMA(0, 1, At, B1); BAR;
        LDA(At, 0, 1); STAGE(SA(0, 0), a2);
        BAR; WAIT_L(0); MMA(1, 0, At, B0); BAR; SCHED;
        STAGE(SB(0, 1), b2 + HTB);
        WAIT_V(6); BAR; MMA(1, 1, At, B1); BAR;
        RESCALE(t + 1);
        LDB(B0, 1, 0); SCHED; LDA(At, 1, 0); STAGE(SA(0, 1), a2 + HTB);
        WAIT_L(8); BAR; WAIT_L(0); MMA(0, 0, At, B0); BAR; SCHED;
        LDB(B1, 1, 1); STAGE(SB(1, 0), b3);
        BAR; WAIT_L(0); MMA(0, 1, At, B1); BAR;
        LDA(At, 1, 1); STAGE(SA(1, 0), a3);
        BAR; WAIT_L(0); MMA(1, 0, At, B0); BAR; SCHED;
        STAGE(SB(1, 1), b3 + HTB);
        WAIT_V(6); BAR; MMA(1, 1, At, B1); BAR;
    }
    WAIT_V(0);
    if (wr == 0) BAR;
    BAR;
    int tz = threadIdx.x; asm volatile("" : "+v"(tz));
    const int lane2 = tz & 63, fr2 = lane2 & 15, fq2 = lane2 >> 4;
    const bool bad = bad_in || (__hip_atomic_load(ctl(myws, L_TMO), __ATOMIC_RELAXED, __HIP_MEMORY_SCOPE_AGENT) == call);
    const int owner = mt >> 1;
    float* obase = p.out[owner] + (size_t)((mt & 1) * BM) * NG + (size_t)p.rank * NS + nt * BM;
    const float qnan = __builtin_nanf("");
#pragma unroll
    for (int ai = 0; ai < 2; ++ai) {
#pragma unroll
        for (int m = 0; m < 4; ++m) {
            const int r = wr * 64 + m * 16 + fr2;
#pragma unroll
            for (int bj = 0; bj < 2; ++bj)
#pragma unroll
                for (int n = 0; n < 2; ++n) {
                    f32x4 a = acc[ai][bj][m][n], o;
#pragma unroll
                    for (int j = 0; j < 4; ++j) o[j] = bad ? qnan : silu_f(ldexpf(a[j], -ecur));
                    const int ch = bj * 32 + wc * 8 + n * 4 + fq2;
                    *(LAS f32x4*)(lds + r * 1024 + ((ch ^ (r & 7)) << 4)) = o;
                }
        }
        __syncthreads();
#pragma unroll 4
        for (int i = 0; i < 16; ++i) {
            const int r = wid * 16 + i;
            f32x4 o = *(LAS f32x4*)(lds + r * 1024 + ((lane2 ^ (r & 7)) << 4));
            *(f32x4*)(obase + (size_t)(ai * HALF + r) * NG + lane2 * 4) = o;
        }
        __syncthreads();
    }
    wg_release();
    if (threadIdx.x == 0) sys_add(ctl(p.ws[owner], L_DONE), 1u);
#undef SA
#undef SB
#undef STAGE
#undef LDA
#undef LDB
#undef MMA
#undef RESCALE
}

}

__global__ void __launch_bounds__(NTHR, 2) fused_kernel(int phase, Params p) {
    extern __shared__ __attribute__((aligned(16))) unsigned char smem[];
    LAS unsigned char* lds = (LAS unsigned char*)smem;
    char* myws = p.ws[p.rank];
    unsigned* tmo = ctl(myws, L_TMO);
    if (phase == 0) {
        const unsigned call = call_number(myws, L_TICK0, gridDim.x, (LAS unsigned*)(lds + LDS_BYTES - 128));
        const int par = call & 1;
        for (int u = blockIdx.x; u < 16 * 72; u += gridDim.x) {
            const int j = u / 72, i = u % 72;
            if (i < 8) produce_x(p, par, j, i, lds);
            else { const int v = i - 8; produce_w(p, par, (p.rank + 1 + (v >> 3)) & 7, j, v & 7, lds); }
            __syncthreads();
        }
        wg_release();
        if (threadIdx.x == 0)
            for (int d = 0; d < NR; ++d) sys_add(ctl(p.ws[(p.rank + d) & 7], L_INRDY), 1u);
    } else if (phase == 1) {
        const unsigned call = call_number(myws, L_TICK1, gridDim.x, (LAS unsigned*)(lds + LDS_BYTES - 128));
        const int par = call & 1;
        if (threadIdx.x == 0) {
            wait_flag(ctl(myws, L_INRDY), call * (unsigned)(NR * GRID), tmo, call);
            __builtin_amdgcn_fence(__ATOMIC_ACQUIRE, "");
            asm volatile("s_waitcnt vmcnt(0)" ::: "memory");
        }
        __syncthreads();
        const int tl = p.tile_of_block[blockIdx.x];
        gemm_tile(p, par, tl >> 4, tl & 15, call, lds, false);
    } else {
        const unsigned call = call_number(myws, L_TICK2, gridDim.x, (LAS unsigned*)(lds + LDS_BYTES - 128));
        if (threadIdx.x == 0) {
            wait_flag(ctl(myws, L_DONE), call * (unsigned)(NR * 2 * NNT), tmo, call);
            __builtin_amdgcn_fence(__ATOMIC_ACQUIRE, "");
            if (__hip_atomic_load(tmo, __ATOMIC_RELAXED, __HIP_MEMORY_SCOPE_AGENT) == call)
                p.out[p.rank][(size_t)p.rank * NS] = __builtin_nanf("");
        }
    }
}

extern "C" void kernel_launch(int rank, int nranks, ncclComm_t comm,
                              hipStream_t stream,
                              void* const* d_in, const int* in_sizes, int n_in,
                              void* d_out, int out_size,
                              const int* in_dtypes, int out_dtype,
                              void* const* d_out_all,
                              void* const* d_ws_all, size_t ws_bytes) {
    if (nranks != NR || ws_bytes < WS_NEED || n_in < 2) { fprintf(stderr, "unexpected configuration\n"); return; }
    static bool attr_set = false;
    if (!attr_set) {
        HIP_CHECK(hipFuncSetAttribute((const void*)fused_kernel, hipFuncAttributeMaxDynamicSharedMemorySize, LDS_BYTES));
        attr_set = true;
    }
    Params p;
    p.x = (const float*)d_in[0];
    p.w = (const float*)d_in[1];
    for (int i = 0; i < NR; ++i) { p.out[i] = (float*)d_out_all[i]; p.ws[i] = (char*)d_ws_all[i]; }
    p.rank = rank;
    p.nprod = 0;
    for (int b = 0; b < GRID; ++b) {
        const int x = b & 7, lb = b >> 3, a = x & 3, c = x >> 2;
        const int mt = a + 4 * (lb & 3), nt = c + 2 * (lb >> 2);
        p.tile_of_block[b] = (unsigned char)((mt << 4) | nt);
    }
    fused_kernel<<<GRID, NTHR, LDS_BYTES, stream>>>(0, p);
    fused_kernel<<<GRID, NTHR, LDS_BYTES, stream>>>(1, p);
    fused_kernel<<<1, NTHR, LDS_BYTES, stream>>>(2, p);
    HIP_CHECK(hipGetLastError());
}
```

```cpp
#include <hip/hip_runtime.h>
#include <rccl/rccl.h>
#include <cstdio>
#include <cstdint>

#define HIP_CHECK(expr) do { hipError_t _e = (expr); if (_e != hipSuccess) { \
    fprintf(stderr, "HIP error %s at %s:%d\n", hipGetErrorString(_e), __FILE__, __LINE__); \
    return; } } while (0)

#define LAS __attribute__((address_space(3)))
typedef _Float16 h8 __attribute__((ext_vector_type(8)));
typedef float f32x4 __attribute__((ext_vector_type(4)));
typedef unsigned u32x4 __attribute__((ext_vector_type(4)));

namespace {

constexpr int NR = 8;
constexpr int MG = 4096, KG = 4096, NG = 32768;
constexpr int KS = KG / NR;
constexpr int NS = NG / NR;
constexpr int BM = 256, BK = 64, HALF = 128;
constexpr int NMT = MG / BM;
constexpr int NNT = NS / BM;
constexpr int NKT = KG / BK;
constexpr int KTS = KS / BK;
constexpr int HTB = HALF * BK * 2;
constexpr int TB = 2 * HTB;
constexpr int LDS_BYTES = 8 * HTB;
constexpr int NTHR = 512;
constexpr int GRID = 256;
#define SLEEPN 32
#define THROT 1
#define NPROD 64

constexpr size_t L_TICK0 = 0 * 128, L_TICK1 = 1 * 128, L_TICK2 = 2 * 128, L_TMO = 3 * 128, L_DONE = 4 * 128, L_INRDY = 5 * 128;
constexpr size_t L_XRDY = 16 * 128;
constexpr size_t L_WRDY = 32 * 128;
constexpr size_t O_EX = 64 * 1024;
constexpr size_t O_EW = O_EX + 2 * 16 * 64 * 4;
constexpr size_t O_XS = 1 << 20;
constexpr size_t PANEL = (size_t)NKT * TB;
constexpr size_t OPB = (size_t)16 * PANEL;
constexpr size_t O_WS = O_XS + 2 * OPB;
constexpr size_t WS_NEED = O_WS + 2 * OPB;

#define STAMP(i) do { } while (0)
constexpr unsigned long long DEADLINE = 150000000ull;

struct Params {
    const float* x;
    const float* w;
    float* out[NR];
    char* ws[NR];
    int rank;
    int nprod;
    unsigned char tile_of_block[GRID];
    unsigned char prod_of_block[GRID];
};

__host__ __device__ __forceinline__ int lds_byte(int r, int c) {
    const int st = (r >> 4) * 2 + (c >> 5), rr = r & 15, cc = c & 31, ob = rr * 64 + cc * 2;
    return st * 1024 + (ob ^ (((ob >> 9) & 1) << 5));
}
__host__ __device__ __forceinline__ void stage_rc(int b, int& R, int& C) {
    const int st = b / 1024, sb = b % 1024, swz = sb ^ (((sb >> 9) & 1) << 5);
    R = (st >> 1) * 16 + swz / 64; C = (st & 1) * 32 + (swz % 64) / 2;
}

__device__ __forceinline__ unsigned* ctl(char* ws, size_t off) { return (unsigned*)(ws + off); }

__device__ __forceinline__ unsigned call_number(char* myws, size_t site, unsigned nwg, LAS unsigned* sh) {
    if (threadIdx.x == 0) {
        unsigned t = __hip_atomic_fetch_add(ctl(myws, site), 1u, __ATOMIC_RELAXED, __HIP_MEMORY_SCOPE_AGENT);
        sh[0] = t / nwg + 1;
    }
    __syncthreads();
    unsigned c = sh[0];
    __syncthreads();
    return __builtin_amdgcn_readfirstlane(c);
}

__device__ __forceinline__ void wait_flag(unsigned* flag, unsigned target, unsigned* tmo, unsigned call) {
    unsigned long long t0 = __builtin_amdgcn_s_memrealtime();
    while (__hip_atomic_load(flag, __ATOMIC_RELAXED, __HIP_MEMORY_SCOPE_SYSTEM) < target) {
        if (__hip_atomic_load(tmo, __ATOMIC_RELAXED, __HIP_MEMORY_SCOPE_AGENT) == call
            || __builtin_amdgcn_s_memrealtime() - t0 > DEADLINE) {
            __hip_atomic_store(tmo, call, __ATOMIC_RELAXED, __HIP_MEMORY_SCOPE_AGENT);
            break;
        }
        __builtin_amdgcn_s_sleep(SLEEPN);
    }
}

__device__ __forceinline__ int scale_exp(float amax) {
    unsigned b = __float_as_uint(amax);
    int ex = (int)((b >> 23) & 255);
    if (amax == 0.f || ex == 255) return 0;
    int il = ex - 127;
    int e = ((13 - il) >> 3) << 3;
    e = e > 64 ? 64 : e; e = e < -64 ? -64 : e;
    return e;
}

__device__ __forceinline__ float wg_amax(float v, LAS float* red) {
    unsigned u = __float_as_uint(v);
#pragma unroll
    for (int o = 32; o > 0; o >>= 1) { unsigned t = (unsigned)__shfl_xor((int)u, o); u = u > t ? u : t; }
    const int wid = threadIdx.x >> 6;
    __syncthreads();
    if ((threadIdx.x & 63) == 0) ((LAS unsigned*)red)[wid] = u;
    __syncthreads();
    unsigned m = 0;
#pragma unroll
    for (int i = 0; i < 8; ++i) { unsigned t = ((LAS unsigned*)red)[i]; m = m > t ? m : t; }
    return __uint_as_float(m);
}

__device__ __forceinline__ h8 cvt8(f32x4 a, f32x4 b, int e) {
    h8 r;
    r[0] = (_Float16)ldexpf(a[0], e); r[1] = (_Float16)ldexpf(a[1], e); r[2] = (_Float16)ldexpf(a[2], e); r[3] = (_Float16)ldexpf(a[3], e);
    r[4] = (_Float16)ldexpf(b[0], e); r[5] = (_Float16)ldexpf(b[1], e); r[6] = (_Float16)ldexpf(b[2], e); r[7] = (_Float16)ldexpf(b[3], e);
    return r;
}

__device__ __forceinline__ float abs4max(f32x4 a) {
    unsigned m0 = __float_as_uint(a[0]) & 0x7fffffffu, m1 = __float_as_uint(a[1]) & 0x7fffffffu;
    unsigned m2 = __float_as_uint(a[2]) & 0x7fffffffu, m3 = __float_as_uint(a[3]) & 0x7fffffffu;
    m0 = m0 > m1 ? m0 : m1; m2 = m2 > m3 ? m2 : m3; m0 = m0 > m2 ? m0 : m2;
    return __uint_as_float(m0);
}
__device__ __forceinline__ float umaxf(float a, float b) { return __float_as_uint(a) > __float_as_uint(b) ? a : b; }

__device__ __forceinline__ __amdgpu_buffer_rsrc_t make_rsrc(const void* base) {
    const unsigned long long a = (unsigned long long)base;
    const unsigned lo = __builtin_amdgcn_readfirstlane((unsigned)a), hi = __builtin_amdgcn_readfirstlane((unsigned)(a >> 32));
    return __builtin_amdgcn_make_buffer_rsrc((void*)(((unsigned long long)hi << 32) | lo), 0, 0x7fffffff, 0x00020000);
}
__device__ __forceinline__ void st16_wt(__amdgpu_buffer_rsrc_t r, unsigned off, h8 v) {
    __builtin_amdgcn_raw_buffer_store_b128(__builtin_bit_cast(u32x4, v), r, off, 0, 17);
}
__device__ __forceinline__ void st16_wt(__amdgpu_buffer_rsrc_t r, unsigned off, f32x4 v) {
    __builtin_amdgcn_raw_buffer_store_b128(__builtin_bit_cast(u32x4, v), r, off, 0, 17);
}
struct UnitId { int j, kp, dst; bool isx; };
__device__ __forceinline__ UnitId unit_id(const Params& p, int u) {
    UnitId r; const int pn = u >> 5, i = u & 31;
    r.isx = pn < 8 || (pn >= 14 && pn < 22);
    r.j = pn < 8 ? 2 * pn : pn < 14 ? pn - 8 : pn < 22 ? 2 * (pn - 14) + 1 : pn - 16;
    r.kp = i >> 3;
    r.dst = (p.rank + 1 + (i & 7)) & 7;
    return r;
}
__device__ __forceinline__ int wset(int nt) { return nt < 12 ? nt / 3 : 4 + ((nt - 12) >> 1); }
__device__ __forceinline__ void load_unit(const Params& p, const UnitId& U, f32x4 (&v)[16]) {
    const int tid = threadIdx.x;
    if (U.isx) {
#pragma unroll
        for (int q = 0; q < 8; ++q) {
            const int o = ((q & 3) * NTHR + tid) * 16, half = o >> 14;
            int R, C; stage_rc(o & 16383, R, C);
            const float* src = p.x + (size_t)(U.j * BM + half * HALF + R) * KS + (U.kp * 2 + (q >> 2)) * BK + C;
            v[2 * q] = *(const f32x4*)src; v[2 * q + 1] = *(const f32x4*)(src + 4);
        }
    } else {
        const int wid = tid >> 6, lane = tid & 63;
        const int kh = wid & 1, nb = wid >> 1, kc = lane >> 4, ng = lane & 15;
        const float* src = p.w + (size_t)(U.kp * 2 * BK + kh * 32 + kc * 8) * NG + U.dst * NS + U.j * BM + nb * 64 + 4 * ng;
#pragma unroll
        for (int j = 0; j < 16; ++j) v[j] = __builtin_nontemporal_load((const f32x4*)(src + (size_t)((j >> 3) * BK + (j & 7)) * NG));
    }
}
__device__ __forceinline__ void store_unit(const Params& p, int par, const UnitId& U, const f32x4 (&v)[16], int e, LAS unsigned char* lds) {
    const int tid = threadIdx.x;
    const int kt = p.rank * KTS + 2 * U.kp;
    char* wsd = p.ws[U.dst];
    if (U.isx) {
        __amdgpu_buffer_rsrc_t rs = make_rsrc(wsd + O_XS + (size_t)par * OPB + (size_t)U.j * PANEL + (size_t)kt * TB);
#pragma unroll
        for (int q = 0; q < 8; ++q) st16_wt(rs, (q * NTHR + tid) * 16, cvt8(v[2 * q], v[2 * q + 1], e));
        if (tid == 0) __hip_atomic_store((int*)(wsd + O_EX) + (par * 16 + U.j) * 32 + (kt >> 1), e, __ATOMIC_RELAXED, __HIP_MEMORY_SCOPE_SYSTEM);
    } else {
        const int wid = tid >> 6, lane = tid & 63;
        const int kh = wid & 1, nb = wid >> 1, kc = lane >> 4, ng = lane & 15;
#pragma unroll
        for (int t = 0; t < 2; ++t)
#pragma unroll
            for (int i = 0; i < 4; ++i) {
                f32x4 lo = {v[8 * t + 0][i], v[8 * t + 1][i], v[8 * t + 2][i], v[8 * t + 3][i]}, hi = {v[8 * t + 4][i], v[8 * t + 5][i], v[8 * t + 6][i], v[8 * t + 7][i]};
                const int nl = nb * 64 + 4 * ng + i;
                *(LAS h8*)(lds + t * TB + (nl >> 7) * HTB + lds_byte(nl & 127, kh * 32 + kc * 8)) = cvt8(lo, hi, e);
            }
        __syncthreads();
        __amdgpu_buffer_rsrc_t rs = make_rsrc(wsd + O_WS + (size_t)par * OPB + (size_t)U.j * PANEL + (size_t)kt * TB);
#pragma unroll
        for (int q = 0; q < 8; ++q) {
            h8 t = *(LAS h8*)(lds + (q * NTHR + tid) * 16);
            st16_wt(rs, (q * NTHR + tid) * 16, t);
        }
        if (tid == 0) __hip_atomic_store((int*)(wsd + O_EW) + (par * 16 + U.j) * 32 + (kt >> 1), e, __ATOMIC_RELAXED, __HIP_MEMORY_SCOPE_SYSTEM);
    }
}

__device__ __forceinline__ void wg_release() {
    asm volatile("s_waitcnt vmcnt(0)" ::: "memory");
    __syncthreads();
    if (threadIdx.x == 0) {
        __builtin_amdgcn_fence(__ATOMIC_RELEASE, "");
        asm volatile("s_waitcnt vmcnt(0)" ::: "memory");
    }
}
__device__ __forceinline__ void sys_add(unsigned* ctr, unsigned v) {
    __hip_atomic_fetch_add(ctr, v, __ATOMIC_RELAXED, __HIP_MEMORY_SCOPE_SYSTEM);
}

__device__ __forceinline__ float silu_f(float y) {
    float c = fminf(fmaxf(y, -60.f), 60.f);
    return y / (1.f + __expf(-c));
}

__device__ __forceinline__ void gemm_tile(const Params& p, int par, int mt, int nt, unsigned call, LAS unsigned char* lds, bool bad_in) {
    char* myws = p.ws[p.rank];
    const int tid = threadIdx.x, wid = __builtin_amdgcn_readfirstlane(tid >> 6), lane = tid & 63;
    const int wr = wid >> 2, wc = wid & 3, fr = lane & 15, fq = lane >> 4;
    const char* cA = myws + O_XS + (size_t)par * OPB + (size_t)mt * PANEL;
    const char* cB = myws + O_WS + (size_t)par * OPB + (size_t)nt * PANEL;
    int ev = ((const int*)(myws + O_EX))[(par * 16 + mt) * 32 + (lane & 31)] + ((const int*)(myws + O_EW))[(par * 16 + nt) * 32 + (lane & 31)];
    const unsigned voff = (unsigned)tid * 16u;
    const unsigned ldsw = (unsigned)wid * 1024u;
    const int aoff = lds_byte(wr * 64 + fr, fq * 8), boff = lds_byte(wc * 32 + fr, fq * 8);
#define SA(b, h) (((b) * 2 + (h)) * HTB)
#define SB(b, h) ((4 + (b) * 2 + (h)) * HTB)
#define STAGE(bufoff, gbase) do { _Pragma("unroll") for (int _i = 0; _i < 2; ++_i) \
        __builtin_amdgcn_global_load_lds((const unsigned*)((gbase) + voff + _i * 8192), (LAS unsigned*)(lds + (bufoff) + ldsw + _i * 8192), 16, 0, 0); } while (0)
#define LDA(dst, b, h) do { _Pragma("unroll") for (int m = 0; m < 4; ++m) _Pragma("unroll") for (int k = 0; k < 2; ++k) dst[m][k] = *(const LAS h8*)(lds + SA(b, h) + aoff + m * 2048 + k * 1024); } while (0)
#define LDB(dst, b, h) do { _Pragma("unroll") for (int n = 0; n < 2; ++n) _Pragma("unroll") for (int k = 0; k < 2; ++k) dst[n][k] = *(const LAS h8*)(lds + SB(b, h) + boff + n * 2048 + k * 1024); } while (0)
#define MMA(ai, bj, At, Bt) do { __builtin_amdgcn_s_setprio(1); _Pragma("unroll") for (int m = 0; m < 4; ++m) _Pragma("unroll") for (int n = 0; n < 2; ++n) _Pragma("unroll") for (int k = 0; k < 2; ++k) \
        acc[ai][bj][m][n] = __builtin_amdgcn_mfma_f32_16x16x32_f16(Bt[n][k], At[m][k], acc[ai][bj][m][n], 0, 0, 0); __builtin_amdgcn_s_setprio(0); } while (0)
#define WAIT_V(n) asm volatile("s_waitcnt vmcnt(" #n ")" ::: "memory")
#define WAIT_L(n) asm volatile("s_waitcnt lgkmcnt(" #n ")" ::: "memory")
#define BAR __builtin_amdgcn_s_barrier()
#define SCHED __builtin_amdgcn_sched_barrier(0)
#define RESCALE(tt) do { const int _e = __builtin_amdgcn_readlane(ev, (tt)); if (_e != ecur) { const int _d = _e - ecur; ecur = _e; \
        _Pragma("unroll") for (int a = 0; a < 2; ++a) _Pragma("unroll") for (int b = 0; b < 2; ++b) _Pragma("unroll") for (int m = 0; m < 4; ++m) _Pragma("unroll") for (int n = 0; n < 2; ++n) \
        _Pragma("unroll") for (int j = 0; j < 4; ++j) acc[a][b][m][n][j] = ldexpf(acc[a][b][m][n][j], _d); } } while (0)
    f32x4 acc[2][2][4][2];
#pragma unroll
    for (int a = 0; a < 2; ++a)
#pragma unroll
        for (int b = 0; b < 2; ++b)
#pragma unroll
            for (int m = 0; m < 4; ++m)
#pragma unroll
                for (int n = 0; n < 2; ++n) acc[a][b][m][n] = (f32x4){0.f, 0.f, 0.f, 0.f};
    h8 At[4][2], B0[2][2], B1[2][2];
    int ecur = __builtin_amdgcn_readlane(ev, 0);
    STAGE(SB(0, 0), cB); STAGE(SA(0, 0), cA); STAGE(SB(0, 1), cB + HTB); STAGE(SA(0, 1), cA + HTB);
    if (wr == 1) BAR;
    WAIT_V(4); BAR;
    STAGE(SB(1, 0), cB + TB); STAGE(SA(1, 0), cA + TB); STAGE(SB(1, 1), cB + TB + HTB);
    WAIT_V(6); BAR;
    for (int t = 0; t < NKT; t += 2) {
        const bool last = (t == NKT - 2);
        const char* a1 = cA + (size_t)(t + 1) * TB;
        const char* a2 = last ? cA : cA + (size_t)(t + 2) * TB; const char* b2 = last ? cB : cB + (size_t)(t + 2) * TB;
        const char* a3 = a2 + TB; const char* b3 = b2 + TB;
        RESCALE(t >> 1);
        LDB(B0, 0, 0); SCHED; LDA(At, 0, 0); STAGE(SA(1, 1), a1 + HTB);
        WAIT_L(8); BAR; WAIT_L(0); MMA(0, 0, At, B0); BAR; SCHED;
        LDB(B1, 0, 1); STAGE(SB(0, 0), b2);
        BAR; WAIT_L(0); MMA(0, 1, At, B1); BAR;
        LDA(At, 0, 1); STAGE(SA(0, 0), a2);
        BAR; WAIT_L(0); MMA(1, 0, At, B0); BAR; SCHED;
        STAGE(SB(0, 1), b2 + HTB);
        WAIT_V(6); BAR; MMA(1, 1, At, B1); BAR;
        LDB(B0, 1, 0); SCHED; LDA(At, 1, 0); STAGE(SA(0, 1), a2 + HTB);
        WAIT_L(8); BAR; WAIT_L(0); MMA(0, 0, At, B0); BAR; SCHED;
        LDB(B1, 1, 1); STAGE(SB(1, 0), b3);
        BAR; WAIT_L(0); MMA(0, 1, At, B1); BAR;
        LDA(At, 1, 1); STAGE(SA(1, 0), a3);
        BAR; WAIT_L(0); MMA(1, 0, At, B0); BAR; SCHED;
        STAGE(SB(1, 1), b3 + HTB);
        WAIT_V(6); BAR; MMA(1, 1, At, B1); BAR;
    }
    WAIT_V(0);
    if (wr == 0) BAR;
    BAR;
    STAMP(4);
    int tz = threadIdx.x; asm volatile("" : "+v"(tz));
    const int lane2 = tz & 63, fr2 = lane2 & 15, fq2 = lane2 >> 4;
    const bool bad = bad_in || (__hip_atomic_load(ctl(myws, L_TMO), __ATOMIC_RELAXED, __HIP_MEMORY_SCOPE_AGENT) == call);
    const int owner = mt >> 1;
    float* obase = p.out[owner] + (size_t)((mt & 1) * BM) * NG + (size_t)p.rank * NS + nt * BM;
    const float qnan = __builtin_nanf("");
    __amdgpu_buffer_rsrc_t ors = make_rsrc(obase);
#pragma unroll
    for (int ai = 0; ai < 2; ++ai) {
#pragma unroll
        for (int m = 0; m < 4; ++m) {
            const int r = wr * 64 + m * 16 + fr2;
#pragma unroll
            for (int bj = 0; bj < 2; ++bj)
#pragma unroll
                for (int n = 0; n < 2; ++n) {
                    f32x4 a = acc[ai][bj][m][n], o;
#pragma unroll
                    for (int j = 0; j < 4; ++j) o[j] = bad ? qnan : silu_f(ldexpf(a[j], -ecur));
                    const int ch = bj * 32 + wc * 8 + n * 4 + fq2;
                    *(LAS f32x4*)(lds + r * 1024 + ((ch ^ (r & 7)) << 4)) = o;
                }
        }
        __syncthreads();
#pragma unroll 4
        for (int i = 0; i < 16; ++i) {
            const int r = wid * 16 + i;
            f32x4 o = *(LAS f32x4*)(lds + r * 1024 + ((lane2 ^ (r & 7)) << 4));
            st16_wt(ors, (unsigned)(((ai * HALF + r) * NG + lane2 * 4) * 4), o);
            asm volatile("s_waitcnt vmcnt(%0)" :: "i"(THROT) : "memory");
        }
        __syncthreads();
    }
    STAMP(5);
    asm volatile("s_waitcnt vmcnt(0)" ::: "memory");
    __syncthreads();
    if (threadIdx.x == 0) sys_add(ctl(p.ws[owner], L_DONE), 1u);
    STAMP(6);
#undef SA
#undef SB
#undef STAGE
#undef LDA
#undef LDB
#undef MMA
#undef RESCALE
}

}

__global__ void __launch_bounds__(NTHR, 2) fused_kernel(int phase, Params p) {
    extern __shared__ __attribute__((aligned(16))) unsigned char smem[];
    LAS unsigned char* lds = (LAS unsigned char*)smem;
    char* myws = p.ws[p.rank];
    unsigned* tmo = ctl(myws, L_TMO);
    if (phase == 0) {
        const unsigned call = call_number(myws, L_TICK0, gridDim.x, (LAS unsigned*)(lds + LDS_BYTES - 128));
        const int par = call & 1;
        if (threadIdx.x == 0) { __builtin_amdgcn_fence(__ATOMIC_ACQUIRE, ""); asm volatile("s_waitcnt vmcnt(0)" ::: "memory"); }
        __syncthreads();
        STAMP(0);
        const int b = blockIdx.x;
        const int pidx = p.prod_of_block[b] - 1;
        if (pidx >= 0) {
            f32x4 va[16], vb[16];
            int u = pidx;
            UnitId UA = unit_id(p, u), UB = UA;
            load_unit(p, UA, va);
            unsigned* fprev = nullptr;
#define PSTEP(UC, vc, UN, vn) { \
                const bool more = (u + p.nprod) < 1024; \
                if (more) { UN = unit_id(p, u + p.nprod); load_unit(p, UN, vn); } \
                if (fprev) { \
                    if (more) asm volatile("s_waitcnt vmcnt(16)" ::: "memory"); else asm volatile("s_waitcnt vmcnt(0)" ::: "memory"); \
                    if ((threadIdx.x & 63) == 0) sys_add(fprev, 1u); \
                } \
                float am = 0.f; \
                _Pragma("unroll") for (int q = 0; q < 16; ++q) am = umaxf(am, abs4max(vc[q])); \
                am = wg_amax(am, (LAS float*)(lds + LDS_BYTES - 64)); \
                store_unit(p, par, UC, vc, scale_exp(am), lds); \
                fprev = ctl(p.ws[UC.dst], UC.isx ? L_XRDY + (UC.j & 1) * 128 : L_WRDY + wset(UC.j) * 128); \
                u += p.nprod; \
                if (!more) break; }
            while (true) {
                PSTEP(UA, va, UB, vb)
                PSTEP(UB, vb, UA, va)
            }
#undef PSTEP
            asm volatile("s_waitcnt vmcnt(0)" ::: "memory");
            if ((threadIdx.x & 63) == 0) sys_add(fprev, 1u);
            STAMP(1);
        }
        const int tl = p.tile_of_block[b], mt = tl >> 4, nt = tl & 15;
        STAMP(2);
        if (threadIdx.x == 0) {
            wait_flag(ctl(myws, L_XRDY + (mt & 1) * 128), call * 2048u, tmo, call);
            wait_flag(ctl(myws, L_WRDY + wset(nt) * 128), call * (nt < 12 ? 768u : 512u), tmo, call);
            __builtin_amdgcn_fence(__ATOMIC_ACQUIRE, "agent");
            asm volatile("s_waitcnt vmcnt(0)" ::: "memory");
        }
        __syncthreads();
        STAMP(3);
        gemm_tile(p, par, mt, nt, call, lds, false);
    } else {
        const unsigned call = call_number(myws, L_TICK2, gridDim.x, (LAS unsigned*)(lds + LDS_BYTES - 128));
        STAMP(7);
        if (threadIdx.x == 0) {
            wait_flag(ctl(myws, L_DONE), call * (unsigned)(NR * 2 * NNT), tmo, call);
            __builtin_amdgcn_fence(__ATOMIC_ACQUIRE, "");
            if (__hip_atomic_load(tmo, __ATOMIC_RELAXED, __HIP_MEMORY_SCOPE_AGENT) == call)
                p.out[p.rank][(size_t)p.rank * NS] = __builtin_nanf("");
        }
        STAMP(8);
    }
}

extern "C" void kernel_launch(int rank, int nranks, ncclComm_t comm,
                              hipStream_t stream,
                              void* const* d_in, const int* in_sizes, int n_in,
                              void* d_out, int out_size,
                              const int* in_dtypes, int out_dtype,
                              void* const* d_out_all,
                              void* const* d_ws_all, size_t ws_bytes) {
    if (nranks != NR || ws_bytes < WS_NEED || n_in < 2) { fprintf(stderr, "unexpected configuration\n"); return; }
    static bool attr_set = false;
    if (!attr_set) {
        HIP_CHECK(hipFuncSetAttribute((const void*)fused_kernel, hipFuncAttributeMaxDynamicSharedMemorySize, LDS_BYTES));
        attr_set = true;
    }
    Params p;
    p.x = (const float*)d_in[0];
    p.w = (const float*)d_in[1];
    for (int i = 0; i < NR; ++i) { p.out[i] = (float*)d_out_all[i]; p.ws[i] = (char*)d_ws_all[i]; }
    p.rank = rank;
    p.nprod = NPROD;
    static const int eg[8] = {0, 0, 1, 1, 0, 1, 0, 1}, eT[8] = {0, 1, 0, 1, 2, 2, 3, 3};
    for (int b = 0; b < GRID; ++b) {
        const int x = b & 7, lb = b >> 3;
        int mt, nt;
        if (lb < 8) { mt = (x & 3) + 4 * (lb & 3); nt = 12 + 2 * (x >> 2) + (lb >> 2); }
        else { const int e = lb - 8; mt = 2 * (e & 7) + eg[x]; nt = 3 * eT[x] + (e >> 3); }
        p.tile_of_block[b] = (unsigned char)((mt << 4) | nt);
        p.prod_of_block[b] = lb < 8 ? (unsigned char)(1 + lb * 8 + x) : 0;
    }
    fused_kernel<<<GRID, NTHR, LDS_BYTES, stream>>>(0, p);
    fused_kernel<<<1, NTHR, LDS_BYTES, stream>>>(2, p);
    HIP_CHECK(hipGetLastError());
}
```

```cpp
#include <hip/hip_runtime.h>
#include <rccl/rccl.h>
#include <cstdio>
#include <cstdint>

#define HIP_CHECK(expr) do { hipError_t _e = (expr); if (_e != hipSuccess) { \
    fprintf(stderr, "HIP error %s at %s:%d\n", hipGetErrorString(_e), __FILE__, __LINE__); \
    return; } } while (0)

#define LAS __attribute__((address_space(3)))
typedef _Float16 h8 __attribute__((ext_vector_type(8)));
typedef float f32x4 __attribute__((ext_vector_type(4)));
typedef unsigned u32x4 __attribute__((ext_vector_type(4)));

namespace {

constexpr int NR = 8;
constexpr int MG = 4096, KG = 4096, NG = 32768;
constexpr int KS = KG / NR;
constexpr int NS = NG / NR;
constexpr int BM = 256, BK = 64, HALF = 128;
constexpr int NMT = MG / BM;
constexpr int NNT = NS / BM;
constexpr int NKT = KG / BK;
constexpr int KTS = KS / BK;
constexpr int HTB = HALF * BK * 2;
constexpr int TB = 2 * HTB;
constexpr int LDS_BYTES = 8 * HTB;
constexpr int NTHR = 512;
constexpr int GRID = 256;
#define SLEEPN 32
#define THR_1 0
#define THR_2 0
#define THR_LATE 0
#define NPROD 64

constexpr size_t L_TICK0 = 0 * 128, L_TICK1 = 1 * 128, L_TICK2 = 2 * 128, L_TMO = 3 * 128, L_DONE = 4 * 128, L_INRDY = 5 * 128, L_PDONE = 6 * 128, L_FIN = 7 * 128;
constexpr size_t L_XRDY = 16 * 128;
constexpr size_t L_WRDY = 32 * 128;
constexpr size_t O_EX = 64 * 1024;
constexpr size_t O_EW = O_EX + 2 * 16 * 64 * 4;
constexpr size_t O_XS = 1 << 20;
constexpr size_t PANEL = (size_t)NKT * TB;
constexpr size_t OPB = (size_t)16 * PANEL;
constexpr size_t O_WS = O_XS + 2 * OPB;
constexpr size_t WS_NEED = O_WS + 2 * OPB;

#define STAMP(i) do { } while (0)
constexpr unsigned long long DEADLINE = 150000000ull;

struct Params {
    const float* x;
    const float* w;
    float* out[NR];
    char* ws[NR];
    int rank;
    int nprod;
    unsigned char tile_of_block[GRID];
    unsigned char prod_of_block[GRID];
};

__host__ __device__ __forceinline__ int lds_byte(int r, int c) {
    const int st = (r >> 4) * 2 + (c >> 5), rr = r & 15, cc = c & 31, ob = rr * 64 + cc * 2;
    return st * 1024 + (ob ^ (((ob >> 9) & 1) << 5));
}
__host__ __device__ __forceinline__ void stage_rc(int b, int& R, int& C) {
    const int st = b / 1024, sb = b % 1024, swz = sb ^ (((sb >> 9) & 1) << 5);
    R = (st >> 1) * 16 + swz / 64; C = (st & 1) * 32 + (swz % 64) / 2;
}

__device__ __forceinline__ unsigned* ctl(char* ws, size_t off) { return (unsigned*)(ws + off); }

__device__ __forceinline__ unsigned call_number(char* myws, size_t site, unsigned nwg, LAS unsigned* sh) {
    if (threadIdx.x == 0) {
        unsigned t = __hip_atomic_fetch_add(ctl(myws, site), 1u, __ATOMIC_RELAXED, __HIP_MEMORY_SCOPE_AGENT);
        sh[0] = t / nwg + 1;
    }
    __syncthreads();
    unsigned c = sh[0];
    __syncthreads();
    return __builtin_amdgcn_readfirstlane(c);
}

__device__ __forceinline__ void wait_flag(unsigned* flag, unsigned target, unsigned* tmo, unsigned call) {
    unsigned long long t0 = __builtin_amdgcn_s_memrealtime();
    while (__hip_atomic_load(flag, __ATOMIC_RELAXED, __HIP_MEMORY_SCOPE_SYSTEM) < target) {
        if (__hip_atomic_load(tmo, __ATOMIC_RELAXED, __HIP_MEMORY_SCOPE_AGENT) == call
            || __builtin_amdgcn_s_memrealtime() - t0 > DEADLINE) {
            __hip_atomic_store(tmo, call, __ATOMIC_RELAXED, __HIP_MEMORY_SCOPE_AGENT);
            break;
        }
        __builtin_amdgcn_s_sleep(SLEEPN);
    }
}

__device__ __forceinline__ int scale_exp(float amax) {
    unsigned b = __float_as_uint(amax);
    int ex = (int)((b >> 23) & 255);
    if (amax == 0.f || ex == 255) return 0;
    int il = ex - 127;
    int e = ((13 - il) >> 3) << 3;
    e = e > 64 ? 64 : e; e = e < -64 ? -64 : e;
    return e;
}

__device__ __forceinline__ float wg_amax(float v, LAS float* red) {
    unsigned u = __float_as_uint(v);
#pragma unroll
    for (int o = 32; o > 0; o >>= 1) { unsigned t = (unsigned)__shfl_xor((int)u, o); u = u > t ? u : t; }
    const int wid = threadIdx.x >> 6;
    __syncthreads();
    if ((threadIdx.x & 63) == 0) ((LAS unsigned*)red)[wid] = u;
    __syncthreads();
    unsigned m = 0;
#pragma unroll
    for (int i = 0; i < 8; ++i) { unsigned t = ((LAS unsigned*)red)[i]; m = m > t ? m : t; }
    return __uint_as_float(m);
}

__device__ __forceinline__ h8 cvt8(f32x4 a, f32x4 b, int e) {
    h8 r;
    r[0] = (_Float16)ldexpf(a[0], e); r[1] = (_Float16)ldexpf(a[1], e); r[2] = (_Float16)ldexpf(a[2], e); r[3] = (_Float16)ldexpf(a[3], e);
    r[4] = (_Float16)ldexpf(b[0], e); r[5] = (_Float16)ldexpf(b[1], e); r[6] = (_Float16)ldexpf(b[2], e); r[7] = (_Float16)ldexpf(b[3], e);
    return r;
}

__device__ __forceinline__ float abs4max(f32x4 a) {
    unsigned m0 = __float_as_uint(a[0]) & 0x7fffffffu, m1 = __float_as_uint(a[1]) & 0x7fffffffu;
    unsigned m2 = __float_as_uint(a[2]) & 0x7fffffffu, m3 = __float_as_uint(a[3]) & 0x7fffffffu;
    m0 = m0 > m1 ? m0 : m1; m2 = m2 > m3 ? m2 : m3; m0 = m0 > m2 ? m0 : m2;
    return __uint_as_float(m0);
}
__device__ __forceinline__ float umaxf(float a, float b) { return __float_as_uint(a) > __float_as_uint(b) ? a : b; }

__device__ __forceinline__ __amdgpu_buffer_rsrc_t make_rsrc(const void* base) {
    const unsigned long long a = (unsigned long long)base;
    const unsigned lo = __builtin_amdgcn_readfirstlane((unsigned)a), hi = __builtin_amdgcn_readfirstlane((unsigned)(a >> 32));
    return __builtin_amdgcn_make_buffer_rsrc((void*)(((unsigned long long)hi << 32) | lo), 0, 0x7fffffff, 0x00020000);
}
__device__ __forceinline__ void st16_wt(__amdgpu_buffer_rsrc_t r, unsigned off, h8 v) {
    __builtin_amdgcn_raw_buffer_store_b128(__builtin_bit_cast(u32x4, v), r, off, 0, 17);
}
__device__ __forceinline__ void st16_wt(__amdgpu_buffer_rsrc_t r, unsigned off, f32x4 v) {
    __builtin_amdgcn_raw_buffer_store_b128(__builtin_bit_cast(u32x4, v), r, off, 0, 17);
}
struct UnitId { int j, kp, dst; bool isx; };
__device__ __forceinline__ UnitId unit_id(const Params& p, int u) {
    UnitId r; const int pn = u >> 5, i = u & 31;
    r.isx = pn < 4 || (pn >= 7 && pn < 11) || (pn >= 14 && pn < 22);
    r.j = pn < 4 ? 4 * pn : pn < 7 ? pn - 4 : pn < 11 ? 4 * (pn - 7) + 2 : pn < 14 ? pn - 8 : pn < 22 ? 4 * ((pn - 14) >> 1) + 1 + 2 * (pn & 1) : pn - 16;
    r.kp = i >> 3;
    r.dst = (p.rank + 1 + (i & 7)) & 7;
    return r;
}
__device__ __forceinline__ int wset(int nt) { return nt < 12 ? nt / 3 : 4 + ((nt - 12) >> 1); }
__device__ __forceinline__ void load_unit(const Params& p, const UnitId& U, f32x4 (&v)[16]) {
    const int tid = threadIdx.x;
    if (U.isx) {
#pragma unroll
        for (int q = 0; q < 8; ++q) {
            const int o = ((q & 3) * NTHR + tid) * 16, half = o >> 14;
            int R, C; stage_rc(o & 16383, R, C);
            const float* src = p.x + (size_t)(U.j * BM + half * HALF + R) * KS + (U.kp * 2 + (q >> 2)) * BK + C;
            v[2 * q] = *(const f32x4*)src; v[2 * q + 1] = *(const f32x4*)(src + 4);
        }
    } else {
        const int wid = tid >> 6, lane = tid & 63;
        const int kh = wid & 1, nb = wid >> 1, kc = lane >> 4, ng = lane & 15;
        const float* src = p.w + (size_t)(U.kp * 2 * BK + kh * 32 + kc * 8) * NG + U.dst * NS + U.j * BM + nb * 64 + 4 * ng;
#pragma unroll
        for (int j = 0; j < 16; ++j) v[j] = __builtin_nontemporal_load((const f32x4*)(src + (size_t)((j >> 3) * BK + (j & 7)) * NG));
    }
}
__device__ __forceinline__ void store_unit(const Params& p, int par, const UnitId& U, const f32x4 (&v)[16], int e, LAS unsigned char* lds) {
    const int tid = threadIdx.x;
    const int kt = p.rank * KTS + 2 * U.kp;
    char* wsd = p.ws[U.dst];
    if (U.isx) {
        __amdgpu_buffer_rsrc_t rs = make_rsrc(wsd + O_XS + (size_t)par * OPB + (size_t)U.j * PANEL + (size_t)kt * TB);
#pragma unroll
        for (int q = 0; q < 8; ++q) st16_wt(rs, (q * NTHR + tid) * 16, cvt8(v[2 * q], v[2 * q + 1], e));
        if (tid == 0) __hip_atomic_store((int*)(wsd + O_EX) + (par * 16 + U.j) * 32 + (kt >> 1), e, __ATOMIC_RELAXED, __HIP_MEMORY_SCOPE_SYSTEM);
    } else {
        const int wid = tid >> 6, lane = tid & 63;
        const int kh = wid & 1, nb = wid >> 1, kc = lane >> 4, ng = lane & 15;
#pragma unroll
        for (int t = 0; t < 2; ++t)
#pragma unroll
            for (int i = 0; i < 4; ++i) {
                f32x4 lo = {v[8 * t + 0][i], v[8 * t + 1][i], v[8 * t + 2][i], v[8 * t + 3][i]}, hi = {v[8 * t + 4][i], v[8 * t + 5][i], v[8 * t + 6][i], v[8 * t + 7][i]};
                const int nl = nb * 64 + 4 * ng + i;
                *(LAS h8*)(lds + t * TB + (nl >> 7) * HTB + lds_byte(nl & 127, kh * 32 + kc * 8)) = cvt8(lo, hi, e);
            }
        __syncthreads();
        __amdgpu_buffer_rsrc_t rs = make_rsrc(wsd + O_WS + (size_t)par * OPB + (size_t)U.j * PANEL + (size_t)kt * TB);
#pragma unroll
        for (int q = 0; q < 8; ++q) {
            h8 t = *(LAS h8*)(lds + (q * NTHR + tid) * 16);
            st16_wt(rs, (q * NTHR + tid) * 16, t);
        }
        if (tid == 0) __hip_atomic_store((int*)(wsd + O_EW) + (par * 16 + U.j) * 32 + (kt >> 1), e, __ATOMIC_RELAXED, __HIP_MEMORY_SCOPE_SYSTEM);
    }
}

__device__ __forceinline__ void wg_release() {
    asm volatile("s_waitcnt vmcnt(0)" ::: "memory");
    __syncthreads();
    if (threadIdx.x == 0) {
        __builtin_amdgcn_fence(__ATOMIC_RELEASE, "");
        asm volatile("s_waitcnt vmcnt(0)" ::: "memory");
    }
}
__device__ __forceinline__ void sys_add(unsigned* ctr, unsigned v) {
    __hip_atomic_fetch_add(ctr, v, __ATOMIC_RELAXED, __HIP_MEMORY_SCOPE_SYSTEM);
}

__device__ __forceinline__ float silu_f(float y) {
    float c = fminf(fmaxf(y, -60.f), 60.f);
    return y / (1.f + __expf(-c));
}

__device__ __forceinline__ void gemm_tile(const Params& p, int par, int mt, int nt, unsigned call, LAS unsigned char* lds, int thr) {
    const bool bad_in = false;
    char* myws = p.ws[p.rank];
    const int tid = threadIdx.x, wid = __builtin_amdgcn_readfirstlane(tid >> 6), lane = tid & 63;
    const int wr = wid >> 2, wc = wid & 3, fr = lane & 15, fq = lane >> 4;
    const char* cA = myws + O_XS + (size_t)par * OPB + (size_t)mt * PANEL;
    const char* cB = myws + O_WS + (size_t)par * OPB + (size_t)nt * PANEL;
    int ev = ((const int*)(myws + O_EX))[(par * 16 + mt) * 32 + (lane & 31)] + ((const int*)(myws + O_EW))[(par * 16 + nt) * 32 + (lane & 31)];
    const unsigned voff = (unsigned)tid * 16u;
    const unsigned ldsw = (unsigned)wid * 1024u;
    const int aoff = lds_byte(wr * 64 + fr, fq * 8), boff = lds_byte(wc * 32 + fr, fq * 8);
#define SA(b, h) (((b) * 2 + (h)) * HTB)
#define SB(b, h) ((4 + (b) * 2 + (h)) * HTB)
#define STAGE(bufoff, gbase) do { _Pragma("unroll") for (int _i = 0; _i < 2; ++_i) \
        __builtin_amdgcn_global_load_lds((const unsigned*)((gbase) + voff + _i * 8192), (LAS unsigned*)(lds + (bufoff) + ldsw + _i * 8192), 16, 0, 0); } while (0)
#define LDA(dst, b, h) do { _Pragma("unroll") for (int m = 0; m < 4; ++m) _Pragma("unroll") for (int k = 0; k < 2; ++k) dst[m][k] = *(const LAS h8*)(lds + SA(b, h) + aoff + m * 2048 + k * 1024); } while (0)
#define LDB(dst, b, h) do { _Pragma("unroll") for (int n = 0; n < 2; ++n) _Pragma("unroll") for (int k = 0; k < 2; ++k) dst[n][k] = *(const LAS h8*)(lds + SB(b, h) + boff + n * 2048 + k * 1024); } while (0)
#define MMA(ai, bj, At, Bt) do { __builtin_amdgcn_s_setprio(1); _Pragma("unroll") for (int m = 0; m < 4; ++m) _Pragma("unroll") for (int n = 0; n < 2; ++n) _Pragma("unroll") for (int k = 0; k < 2; ++k) \
        acc[ai][bj][m][n] = __builtin_amdgcn_mfma_f32_16x16x32_f16(Bt[n][k], At[m][k], acc[ai][bj][m][n], 0, 0, 0); __builtin_amdgcn_s_setprio(0); } while (0)
#define WAIT_V(n) asm volatile("s_waitcnt vmcnt(" #n ")" ::: "memory")
#define WAIT_L(n) asm volatile("s_waitcnt lgkmcnt(" #n ")" ::: "memory")
#define BAR __builtin_amdgcn_s_barrier()
#define SCHED __builtin_amdgcn_sched_barrier(0)
#define RESCALE(tt) do { const int _e = __builtin_amdgcn_readlane(ev, (tt)); if (_e != ecur) { const int _d = _e - ecur; ecur = _e; \
        _Pragma("unroll") for (int a = 0; a < 2; ++a) _Pragma("unroll") for (int b = 0; b < 2; ++b) _Pragma("unroll") for (int m = 0; m < 4; ++m) _Pragma("unroll") for (int n = 0; n < 2; ++n) \
        _Pragma("unroll") for (int j = 0; j < 4; ++j) acc[a][b][m][n][j] = ldexpf(acc[a][b][m][n][j], _d); } } while (0)
    f32x4 acc[2][2][4][2];
#pragma unroll
    for (int a = 0; a < 2; ++a)
#pragma unroll
        for (int b = 0; b < 2; ++b)
#pragma unroll
            for (int m = 0; m < 4; ++m)
#pragma unroll
                for (int n = 0; n < 2; ++n) acc[a][b][m][n] = (f32x4){0.f, 0.f, 0.f, 0.f};
    h8 At[4][2], B0[2][2], B1[2][2];
    int ecur = __builtin_amdgcn_readlane(ev, 0);
    STAGE(SB(0, 0), cB); STAGE(SA(0, 0), cA); STAGE(SB(0, 1), cB + HTB); STAGE(SA(0, 1), cA + HTB);
    if (wr == 1) BAR;
    WAIT_V(4); BAR;
    STAGE(SB(1, 0), cB + TB); STAGE(SA(1, 0), cA + TB); STAGE(SB(1, 1), cB + TB + HTB);
    WAIT_V(6); BAR;
    for (int t = 0; t < NKT; t += 2) {
        const bool last = (t == NKT - 2);
        const char* a1 = cA + (size_t)(t + 1) * TB;
        const char* a2 = last ? cA : cA + (size_t)(t + 2) * TB; const char* b2 = last ? cB : cB + (size_t)(t + 2) * TB;
        const char* a3 = a2 + TB; const char* b3 = b2 + TB;
        RESCALE(t >> 1);
        LDB(B0, 0, 0); SCHED; LDA(At, 0, 0); STAGE(SA(1, 1), a1 + HTB);
        WAIT_L(8); BAR; WAIT_L(0); MMA(0, 0, At, B0); BAR; SCHED;
        LDB(B1, 0, 1); STAGE(SB(0, 0), b2);
        BAR; WAIT_L(0); MMA(0, 1, At, B1); BAR;
        LDA(At, 0, 1); STAGE(SA(0, 0), a2);
        BAR; WAIT_L(0); MMA(1, 0, At, B0); BAR; SCHED;
        STAGE(SB(0, 1), b2 + HTB);
        WAIT_V(6); BAR; MMA(1, 1, At, B1); BAR;
        LDB(B0, 1, 0); SCHED; LDA(At, 1, 0); STAGE(SA(0, 1), a2 + HTB);
        WAIT_L(8); BAR; WAIT_L(0); MMA(0, 0, At, B0); BAR; SCHED;
        LDB(B1, 1, 1); STAGE(SB(1, 0), b3);
        BAR; WAIT_L(0); MMA(0, 1, At, B1); BAR;
        LDA(At, 1, 1); STAGE(SA(1, 0), a3);
        BAR; WAIT_L(0); MMA(1, 0, At, B0); BAR; SCHED;
        STAGE(SB(1, 1), b3 + HTB);
        WAIT_V(6); BAR; MMA(1, 1, At, B1); BAR;
    }
    WAIT_V(0);
    if (wr == 0) BAR;
    BAR;
    STAMP(4);
    int tz = threadIdx.x; asm volatile("" : "+v"(tz));
    const int lane2 = tz & 63, fr2 = lane2 & 15, fq2 = lane2 >> 4;
    const bool bad = bad_in || (__hip_atomic_load(ctl(myws, L_TMO), __ATOMIC_RELAXED, __HIP_MEMORY_SCOPE_AGENT) == call);
    const int owner = mt >> 1;
    float* obase = p.out[owner] + (size_t)((mt & 1) * BM) * NG + (size_t)p.rank * NS + nt * BM;
    const float qnan = __builtin_nanf("");
    __amdgpu_buffer_rsrc_t ors = make_rsrc(obase);
#pragma unroll
    for (int ai = 0; ai < 2; ++ai) {
#pragma unroll
        for (int m = 0; m < 4; ++m) {
            const int r = wr * 64 + m * 16 + fr2;
#pragma unroll
            for (int bj = 0; bj < 2; ++bj)
#pragma unroll
                for (int n = 0; n < 2; ++n) {
                    f32x4 a = acc[ai][bj][m][n], o;
#pragma unroll
                    for (int j = 0; j < 4; ++j) o[j] = bad ? qnan : silu_f(ldexpf(a[j], -ecur));
                    const int ch = bj * 32 + wc * 8 + n * 4 + fq2;
                    *(LAS f32x4*)(lds + r * 1024 + ((ch ^ (r & 7)) << 4)) = o;
                }
        }
        if (ai == 0 && tz == 0) wait_flag(ctl(myws, L_PDONE), call * (unsigned)p.nprod, ctl(myws, L_TMO), call);
        __syncthreads();
#pragma unroll 4
        for (int i = 0; i < 16; ++i) {
            const int r = wid * 16 + i;
            f32x4 o = *(LAS f32x4*)(lds + r * 1024 + ((lane2 ^ (r & 7)) << 4));
            st16_wt(ors, (unsigned)(((ai * HALF + r) * NG + lane2 * 4) * 4), o);
            if (thr == 0) asm volatile("s_waitcnt vmcnt(0)" ::: "memory");
            else if (thr == 1) asm volatile("s_waitcnt vmcnt(1)" ::: "memory");
            else if (thr == 2) asm volatile("s_waitcnt vmcnt(2)" ::: "memory");
            else if (thr == 3) asm volatile("s_waitcnt vmcnt(3)" ::: "memory");
        }
        __syncthreads();
    }
    STAMP(5);
    asm volatile("s_waitcnt vmcnt(0)" ::: "memory");
    __syncthreads();
    if (threadIdx.x == 0) sys_add(ctl(p.ws[owner], L_DONE), 1u);
    STAMP(6);
#undef SA
#undef SB
#undef STAGE
#undef LDA
#undef LDB
#undef MMA
#undef RESCALE
}

}

__global__ void __launch_bounds__(NTHR, 2) fused_kernel(int phase, Params p) {
    extern __shared__ __attribute__((aligned(16))) unsigned char smem[];
    LAS unsigned char* lds = (LAS unsigned char*)smem;
    char* myws = p.ws[p.rank];
    unsigned* tmo = ctl(myws, L_TMO);
    if (phase == 0) {
        const int b = blockIdx.x;
        f32x4 va[16], vb[16];
        UnitId UA = unit_id(p, b), UB = UA;
        load_unit(p, UA, va);
        const unsigned call = call_number(myws, L_TICK0, gridDim.x, (LAS unsigned*)(lds + LDS_BYTES - 128));
        const int par = call & 1;
        if (threadIdx.x == 0) { __builtin_amdgcn_fence(__ATOMIC_ACQUIRE, ""); asm volatile("s_waitcnt vmcnt(0)" ::: "memory"); }
        __syncthreads();
        STAMP(0);
        const int pidx = p.prod_of_block[b] - 1;
        {
            int unext = pidx >= 0 ? GRID + pidx : 1024;
            unsigned* fprev = nullptr;
#define PSTEP(UC, vc, UN, vn) { \
                const bool more = unext < 1024; \
                if (more) { UN = unit_id(p, unext); load_unit(p, UN, vn); } \
                float am = 0.f; \
                _Pragma("unroll") for (int q = 0; q < 16; ++q) am = umaxf(am, abs4max(vc[q])); \
                am = wg_amax(am, (LAS float*)(lds + LDS_BYTES - 64)); \
                store_unit(p, par, UC, vc, scale_exp(am), lds); \
                if (fprev) { \
                    if (more) asm volatile("s_waitcnt vmcnt(24)" ::: "memory"); else asm volatile("s_waitcnt vmcnt(8)" ::: "memory"); \
                    if ((threadIdx.x & 63) == 0) sys_add(fprev, 1u); \
                } \
                fprev = ctl(p.ws[UC.dst], UC.isx ? L_XRDY + (UC.j & 3) * 128 : L_WRDY + wset(UC.j) * 128); \
                unext += p.nprod; \
                if (!more) break; }
            while (true) {
                PSTEP(UA, va, UB, vb)
                PSTEP(UB, vb, UA, va)
            }
#undef PSTEP
            asm volatile("s_waitcnt vmcnt(0)" ::: "memory");
            if ((threadIdx.x & 63) == 0) sys_add(fprev, 1u);
            if (pidx >= 0 && threadIdx.x == 0) __hip_atomic_fetch_add(ctl(myws, L_PDONE), 1u, __ATOMIC_RELAXED, __HIP_MEMORY_SCOPE_AGENT);
            __syncthreads();
            STAMP(1);
        }
        const int tl = p.tile_of_block[b], mt = tl >> 4, nt = tl & 15;
        STAMP(2);
        if (threadIdx.x == 0) {
            wait_flag(ctl(myws, L_XRDY + (mt & 3) * 128), call * 1024u, tmo, call);
            wait_flag(ctl(myws, L_WRDY + wset(nt) * 128), call * (nt < 12 ? 768u : 512u), tmo, call);
            __builtin_amdgcn_fence(__ATOMIC_ACQUIRE, "agent");
            asm volatile("s_waitcnt vmcnt(0)" ::: "memory");
        }
        __syncthreads();
        STAMP(3);
        gemm_tile(p, par, mt, nt, call, lds, (b >> 3) < 8 ? THR_LATE : (b >> 3) < 20 ? THR_1 : THR_2);
        if (threadIdx.x == 0) {
            const unsigned t = __hip_atomic_fetch_add(ctl(myws, L_FIN), 1u, __ATOMIC_RELAXED, __HIP_MEMORY_SCOPE_AGENT);
            if (t + 1 == call * (unsigned)GRID) {
                STAMP(7);
                wait_flag(ctl(myws, L_DONE), call * (unsigned)(NR * 2 * NNT), tmo, call);
                __builtin_amdgcn_fence(__ATOMIC_ACQUIRE, "");
                if (__hip_atomic_load(tmo, __ATOMIC_RELAXED, __HIP_MEMORY_SCOPE_AGENT) == call)
                    p.out[p.rank][(size_t)p.rank * NS] = __builtin_nanf("");
                STAMP(8);
            }
        }
    } else {
        const unsigned call = call_number(myws, L_TICK2, gridDim.x, (LAS unsigned*)(lds + LDS_BYTES - 128));
        STAMP(7);
        if (threadIdx.x == 0) {
            wait_flag(ctl(myws, L_DONE), call * (unsigned)(NR * 2 * NNT), tmo, call);
            __builtin_amdgcn_fence(__ATOMIC_ACQUIRE, "");
            if (__hip_atomic_load(tmo, __ATOMIC_RELAXED, __HIP_MEMORY_SCOPE_AGENT) == call)
                p.out[p.rank][(size_t)p.rank * NS] = __builtin_nanf("");
        }
        STAMP(8);
    }
}

extern "C" void kernel_launch(int rank, int nranks, ncclComm_t comm,
                              hipStream_t stream,
                              void* const* d_in, const int* in_sizes, int n_in,
                              void* d_out, int out_size,
                              const int* in_dtypes, int out_dtype,
                              void* const* d_out_all,
                              void* const* d_ws_all, size_t ws_bytes) {
    if (nranks != NR || ws_bytes < WS_NEED || n_in < 2) { fprintf(stderr, "unexpected configuration\n"); return; }
    static bool attr_set = false;
    if (!attr_set) {
        HIP_CHECK(hipFuncSetAttribute((const void*)fused_kernel, hipFuncAttributeMaxDynamicSharedMemorySize, LDS_BYTES));
        attr_set = true;
    }
    Params p;
    p.x = (const float*)d_in[0];
    p.w = (const float*)d_in[1];
    for (int i = 0; i < NR; ++i) { p.out[i] = (float*)d_out_all[i]; p.ws[i] = (char*)d_ws_all[i]; }
    p.rank = rank;
    p.nprod = NPROD;
    static const int q1[8] = {0, 2, 0, 2, 1, 1, 3, 3}, T1[8] = {0, 0, 1, 1, 0, 1, 0, 1};
    for (int b = 0; b < GRID; ++b) {
        const int x = b & 7, lb = b >> 3;
        int mt, nt;
        if (lb < 8) { mt = (x & 3) + 4 * (lb & 3); nt = 12 + 2 * (x >> 2) + (lb >> 2); }
        else if (lb < 20) { const int e = lb - 8; mt = q1[x] + 4 * (e & 3); nt = 3 * T1[x] + (e >> 2); }
        else { const int e = lb - 20; mt = (x & 3) + 4 * (e & 3); nt = 3 * (2 + (x >> 2)) + (e >> 2); }
        p.tile_of_block[b] = (unsigned char)((mt << 4) | nt);
        p.prod_of_block[b] = lb < 8 ? (unsigned char)(1 + lb * 8 + x) : 0;
    }
    fused_kernel<<<GRID, NTHR, LDS_BYTES, stream>>>(0, p);
    HIP_CHECK(hipGetLastError());
}
```

```cpp
#include <hip/hip_runtime.h>
#include <rccl/rccl.h>
#include <cstdio>
#include <cstdint>

#define HIP_CHECK(expr) do { hipError_t _e = (expr); if (_e != hipSuccess) { \
    fprintf(stderr, "HIP error %s at %s:%d\n", hipGetErrorString(_e), __FILE__, __LINE__); \
    return; } } while (0)

#define LAS __attribute__((address_space(3)))
typedef _Float16 h8 __attribute__((ext_vector_type(8)));
typedef float f32x4 __attribute__((ext_vector_type(4)));
typedef unsigned u32x4 __attribute__((ext_vector_type(4)));

namespace {

constexpr int NR = 8;
constexpr int MG = 4096, KG = 4096, NG = 32768;
constexpr int KS = KG / NR;
constexpr int NS = NG / NR;
constexpr int BM = 256, BK = 64, HALF = 128;
constexpr int NMT = MG / BM;
constexpr int NNT = NS / BM;
constexpr int NKT = KG / BK;
constexpr int KTS = KS / BK;
constexpr int HTB = HALF * BK * 2;
constexpr int TB = 2 * HTB;
constexpr int LDS_BYTES = 8 * HTB;
constexpr int NTHR = 512;
constexpr int GRID = 256;
constexpr int NPROD = 64;
constexpr int LDS_ALLOC = LDS_BYTES + 64;

constexpr size_t L_TICK0 = 0 * 128;
constexpr size_t L_TMO = 3 * 128;
constexpr size_t L_DONE = 4 * 128;
constexpr size_t L_PDONE = 6 * 128;
constexpr size_t L_FIN = 7 * 128;
constexpr size_t L_XRDY = 16 * 128;
constexpr size_t L_WRDY = 32 * 128;
constexpr size_t O_EX = 64 * 1024;
constexpr size_t O_EW = O_EX + 2 * 16 * 64 * 4;
constexpr size_t O_XS = 1 << 20;
constexpr size_t PANEL = (size_t)NKT * TB;
constexpr size_t OPB = (size_t)16 * PANEL;
constexpr size_t O_WS = O_XS + 2 * OPB;
constexpr size_t WS_NEED = O_WS + 2 * OPB;

#define STAMP(i) do { } while (0)
constexpr unsigned long long DEADLINE = 150000000ull;

struct Params {
    const float* x;
    const float* w;
    float* out[NR];
    char* ws[NR];
    int rank;
    unsigned char tile_of_block[GRID];
    unsigned char prod_of_block[GRID];
};

__host__ __device__ __forceinline__ int lds_byte(int r, int c) {
    const int st = (r >> 4) * 2 + (c >> 5), rr = r & 15, cc = c & 31, ob = rr * 64 + cc * 2;
    return st * 1024 + (ob ^ (((ob >> 9) & 1) << 5));
}
__host__ __device__ __forceinline__ void stage_rc(int b, int& R, int& C) {
    const int st = b / 1024, sb = b % 1024, swz = sb ^ (((sb >> 9) & 1) << 5);
    R = (st >> 1) * 16 + swz / 64; C = (st & 1) * 32 + (swz % 64) / 2;
}

__device__ __forceinline__ unsigned* ctl(char* ws, size_t off) { return (unsigned*)(ws + off); }

__device__ __forceinline__ unsigned call_number(char* myws, size_t site, unsigned nwg, LAS unsigned* sh) {
    if (threadIdx.x == 0) {
        unsigned t = __hip_atomic_fetch_add(ctl(myws, site), 1u, __ATOMIC_RELAXED, __HIP_MEMORY_SCOPE_AGENT);
        sh[0] = t / nwg + 1;
    }
    __syncthreads();
    unsigned c = sh[0];
    __syncthreads();
    return __builtin_amdgcn_readfirstlane(c);
}

__device__ __forceinline__ void wait_flag(unsigned* flag, unsigned target, unsigned* tmo, unsigned call) {
    unsigned long long t0 = __builtin_amdgcn_s_memrealtime();
    while (__hip_atomic_load(flag, __ATOMIC_RELAXED, __HIP_MEMORY_SCOPE_SYSTEM) < target) {
        if (__hip_atomic_load(tmo, __ATOMIC_RELAXED, __HIP_MEMORY_SCOPE_AGENT) == call
            || __builtin_amdgcn_s_memrealtime() - t0 > DEADLINE) {
            __hip_atomic_store(tmo, call, __ATOMIC_RELAXED, __HIP_MEMORY_SCOPE_AGENT);
            break;
        }
        __builtin_amdgcn_s_sleep(32);
    }
}

__device__ __forceinline__ int scale_exp(float amax) {
    unsigned b = __float_as_uint(amax);
    int ex = (int)((b >> 23) & 255);
    if (amax == 0.f || ex == 255) return 0;
    int il = ex - 127;
    int e = ((13 - il) >> 3) << 3;
    e = e > 64 ? 64 : e; e = e < -64 ? -64 : e;
    return e;
}

__device__ __forceinline__ float wg_amax(float v, LAS float* red) {
    unsigned u = __float_as_uint(v);
#pragma unroll
    for (int o = 32; o > 0; o >>= 1) { unsigned t = (unsigned)__shfl_xor((int)u, o); u = u > t ? u : t; }
    const int wid = threadIdx.x >> 6;
    __syncthreads();
    if ((threadIdx.x & 63) == 0) ((LAS unsigned*)red)[wid] = u;
    __syncthreads();
    unsigned m = 0;
#pragma unroll
    for (int i = 0; i < 8; ++i) { unsigned t = ((LAS unsigned*)red)[i]; m = m > t ? m : t; }
    return __uint_as_float(m);
}

__device__ __forceinline__ h8 cvt8(f32x4 a, f32x4 b, int e) {
    h8 r;
    r[0] = (_Float16)ldexpf(a[0], e); r[1] = (_Float16)ldexpf(a[1], e); r[2] = (_Float16)ldexpf(a[2], e); r[3] = (_Float16)ldexpf(a[3], e);
    r[4] = (_Float16)ldexpf(b[0], e); r[5] = (_Float16)ldexpf(b[1], e); r[6] = (_Float16)ldexpf(b[2], e); r[7] = (_Float16)ldexpf(b[3], e);
    return r;
}

__device__ __forceinline__ float abs4max(f32x4 a) {
    unsigned m0 = __float_as_uint(a[0]) & 0x7fffffffu, m1 = __float_as_uint(a[1]) & 0x7fffffffu;
    unsigned m2 = __float_as_uint(a[2]) & 0x7fffffffu, m3 = __float_as_uint(a[3]) & 0x7fffffffu;
    m0 = m0 > m1 ? m0 : m1; m2 = m2 > m3 ? m2 : m3; m0 = m0 > m2 ? m0 : m2;
    return __uint_as_float(m0);
}
__device__ __forceinline__ float umaxf(float a, float b) { return __float_as_uint(a) > __float_as_uint(b) ? a : b; }

__device__ __forceinline__ __amdgpu_buffer_rsrc_t make_rsrc(const void* base) {
    const unsigned long long a = (unsigned long long)base;
    const unsigned lo = __builtin_amdgcn_readfirstlane((unsigned)a), hi = __builtin_amdgcn_readfirstlane((unsigned)(a >> 32));
    return __builtin_amdgcn_make_buffer_rsrc((void*)(((unsigned long long)hi << 32) | lo), 0, 0x7fffffff, 0x00020000);
}
__device__ __forceinline__ void st16_wt(__amdgpu_buffer_rsrc_t r, unsigned off, h8 v) {
    __builtin_amdgcn_raw_buffer_store_b128(__builtin_bit_cast(u32x4, v), r, off, 0, 17);
}
__device__ __forceinline__ void st16_wt(__amdgpu_buffer_rsrc_t r, unsigned off, f32x4 v) {
    __builtin_amdgcn_raw_buffer_store_b128(__builtin_bit_cast(u32x4, v), r, off, 0, 17);
}
struct UnitId { int j, kp, dst; bool isx; };
__device__ __forceinline__ UnitId unit_id(const Params& p, int u) {
    UnitId r; const int pn = u >> 5, i = u & 31;
    r.isx = pn < 4 || (pn >= 7 && pn < 11) || (pn >= 14 && pn < 22);
    r.j = pn < 4 ? 4 * pn : pn < 7 ? pn - 4 : pn < 11 ? 4 * (pn - 7) + 2 : pn < 14 ? pn - 8 : pn < 22 ? 4 * ((pn - 14) >> 1) + 1 + 2 * (pn & 1) : pn - 16;
    r.kp = i >> 3;
    r.dst = (p.rank + 1 + (i & 7)) & 7;
    return r;
}
__device__ __forceinline__ int wset(int nt) { return nt < 12 ? nt / 3 : 4 + ((nt - 12) >> 1); }
__device__ __forceinline__ void load_unit(const Params& p, const UnitId& U, f32x4 (&v)[16]) {
    const int tid = threadIdx.x;
    if (U.isx) {
#pragma unroll
        for (int q = 0; q < 8; ++q) {
            const int o = ((q & 3) * NTHR + tid) * 16, half = o >> 14;
            int R, C; stage_rc(o & 16383, R, C);
            const float* src = p.x + (size_t)(U.j * BM + half * HALF + R) * KS + (U.kp * 2 + (q >> 2)) * BK + C;
            v[2 * q] = *(const f32x4*)src; v[2 * q + 1] = *(const f32x4*)(src + 4);
        }
    } else {
        const int wid = tid >> 6, lane = tid & 63;
        const int kh = wid & 1, nb = wid >> 1, kc = lane >> 4, ng = lane & 15;
        const float* src = p.w + (size_t)(U.kp * 2 * BK + kh * 32 + kc * 8) * NG + U.dst * NS + U.j * BM + nb * 64 + 4 * ng;
#pragma unroll
        for (int j = 0; j < 16; ++j) v[j] = __builtin_nontemporal_load((const f32x4*)(src + (size_t)((j >> 3) * BK + (j & 7)) * NG));
    }
}
__device__ __forceinline__ void store_unit(const Params& p, int par, const UnitId& U, const f32x4 (&v)[16], int e, LAS unsigned char* lds) {
    const int tid = threadIdx.x;
    const int kt = p.rank * KTS + 2 * U.kp;
    char* wsd = p.ws[U.dst];
    if (U.isx) {
        __amdgpu_buffer_rsrc_t rs = make_rsrc(wsd + O_XS + (size_t)par * OPB + (size_t)U.j * PANEL + (size_t)kt * TB);
#pragma unroll
        for (int q = 0; q < 8; ++q) st16_wt(rs, (q * NTHR + tid) * 16, cvt8(v[2 * q], v[2 * q + 1], e));
        if (tid == 0) __hip_atomic_store((int*)(wsd + O_EX) + (par * 16 + U.j) * 32 + (kt >> 1), e, __ATOMIC_RELAXED, __HIP_MEMORY_SCOPE_SYSTEM);
    } else {
        const int wid = tid >> 6, lane = tid & 63;
        const int kh = wid & 1, nb = wid >> 1, kc = lane >> 4, ng = lane & 15;
#pragma unroll
        for (int t = 0; t < 2; ++t)
#pragma unroll
            for (int i = 0; i < 4; ++i) {
                f32x4 lo = {v[8 * t + 0][i], v[8 * t + 1][i], v[8 * t + 2][i], v[8 * t + 3][i]}, hi = {v[8 * t + 4][i], v[8 * t + 5][i], v[8 * t + 6][i], v[8 * t + 7][i]};
                const int nl = nb * 64 + 4 * ng + i;
                *(LAS h8*)(lds + t * TB + (nl >> 7) * HTB + lds_byte(nl & 127, kh * 32 + kc * 8)) = cvt8(lo, hi, e);
            }
        __syncthreads();
        __amdgpu_buffer_rsrc_t rs = make_rsrc(wsd + O_WS + (size_t)par * OPB + (size_t)U.j * PANEL + (size_t)kt * TB);
#pragma unroll
        for (int q = 0; q < 8; ++q) {
            h8 t = *(LAS h8*)(lds + (q * NTHR + tid) * 16);
            st16_wt(rs, (q * NTHR + tid) * 16, t);
        }
        if (tid == 0) __hip_atomic_store((int*)(wsd + O_EW) + (par * 16 + U.j) * 32 + (kt >> 1), e, __ATOMIC_RELAXED, __HIP_MEMORY_SCOPE_SYSTEM);
    }
}

__device__ __forceinline__ void sys_add(unsigned* ctr, unsigned v) {
    __hip_atomic_fetch_add(ctr, v, __ATOMIC_RELAXED, __HIP_MEMORY_SCOPE_SYSTEM);
}

__device__ __forceinline__ float silu_f(float y) {
    float c = fminf(fmaxf(y, -60.f), 60.f);
    return y / (1.f + __expf(-c));
}

__device__ __forceinline__ void gemm_tile(const Params& p, int par, int mt, int nt, unsigned call, LAS unsigned char* lds) {
    char* myws = p.ws[p.rank];
    const int tid = threadIdx.x, wid = __builtin_amdgcn_readfirstlane(tid >> 6), lane = tid & 63;
    const int wr = wid >> 2, wc = wid & 3, fr = lane & 15, fq = lane >> 4;
    const char* cA = myws + O_XS + (size_t)par * OPB + (size_t)mt * PANEL;
    const char* cB = myws + O_WS + (size_t)par * OPB + (size_t)nt * PANEL;
    int ev = ((const int*)(myws + O_EX))[(par * 16 + mt) * 32 + (lane & 31)] + ((const int*)(myws + O_EW))[(par * 16 + nt) * 32 + (lane & 31)];
    const unsigned voff = (unsigned)tid * 16u;
    const unsigned ldsw = (unsigned)wid * 1024u;
    const int aoff = lds_byte(wr * 64 + fr, fq * 8), boff = lds_byte(wc * 32 + fr, fq * 8);
#define SA(b, h) (((b) * 2 + (h)) * HTB)
#define SB(b, h) ((4 + (b) * 2 + (h)) * HTB)
#define STAGE(bufoff, gbase) do { _Pragma("unroll") for (int _i = 0; _i < 2; ++_i) \
        __builtin_amdgcn_global_load_lds((const unsigned*)((gbase) + voff + _i * 8192), (LAS unsigned*)(lds + (bufoff) + ldsw + _i * 8192), 16, 0, 0); } while (0)
#define LDA(dst, b, h) do { _Pragma("unroll") for (int m = 0; m < 4; ++m) _Pragma("unroll") for (int k = 0; k < 2; ++k) dst[m][k] = *(const LAS h8*)(lds + SA(b, h) + aoff + m * 2048 + k * 1024); } while (0)
#define LDB(dst, b, h) do { _Pragma("unroll") for (int n = 0; n < 2; ++n) _Pragma("unroll") for (int k = 0; k < 2; ++k) dst[n][k] = *(const LAS h8*)(lds + SB(b, h) + boff + n * 2048 + k * 1024); } while (0)
#define MMA(ai, bj, At, Bt) do { __builtin_amdgcn_s_setprio(1); _Pragma("unroll") for (int m = 0; m < 4; ++m) _Pragma("unroll") for (int n = 0; n < 2; ++n) _Pragma("unroll") for (int k = 0; k < 2; ++k) \
        acc[ai][bj][m][n] = __builtin_amdgcn_mfma_f32_16x16x32_f16(Bt[n][k], At[m][k], acc[ai][bj][m][n], 0, 0, 0); __builtin_amdgcn_s_setprio(0); } while (0)
#define WAIT_V(n) asm volatile("s_waitcnt vmcnt(" #n ")" ::: "memory")
#define WAIT_L(n) asm volatile("s_waitcnt lgkmcnt(" #n ")" ::: "memory")
#define BAR __builtin_amdgcn_s_barrier()
#define SCHED __builtin_amdgcn_sched_barrier(0)
#define RESCALE(tt) do { const int _e = __builtin_amdgcn_readlane(ev, (tt)); if (_e != ecur) { const int _d = _e - ecur; ecur = _e; \
        _Pragma("unroll") for (int a = 0; a < 2; ++a) _Pragma("unroll") for (int b = 0; b < 2; ++b) _Pragma("unroll") for (int m = 0; m < 4; ++m) _Pragma("unroll") for (int n = 0; n < 2; ++n) \
        _Pragma("unroll") for (int j = 0; j < 4; ++j) acc[a][b][m][n][j] = ldexpf(acc[a][b][m][n][j], _d); } } while (0)
    f32x4 acc[2][2][4][2];
#pragma unroll
    for (int a = 0; a < 2; ++a)
#pragma unroll
        for (int b = 0; b < 2; ++b)
#pragma unroll
            for (int m = 0; m < 4; ++m)
#pragma unroll
                for (int n = 0; n < 2; ++n) acc[a][b][m][n] = (f32x4){0.f, 0.f, 0.f, 0.f};
    h8 At[4][2], B0[2][2], B1[2][2];
    int ecur = __builtin_amdgcn_readlane(ev, 0);
    STAGE(SB(0, 0), cB); STAGE(SA(0, 0), cA); STAGE(SB(0, 1), cB + HTB); STAGE(SA(0, 1), cA + HTB);
    if (wr == 1) BAR;
    WAIT_V(4); BAR;
    STAGE(SB(1, 0), cB + TB); STAGE(SA(1, 0), cA + TB); STAGE(SB(1, 1), cB + TB + HTB);
    WAIT_V(6); BAR;
    for (int t = 0; t < NKT; t += 2) {
        const bool last = (t == NKT - 2);
        const char* a1 = cA + (size_t)(t + 1) * TB;
        const char* a2 = last ? cA : cA + (size_t)(t + 2) * TB; const char* b2 = last ? cB : cB + (size_t)(t + 2) * TB;
        const char* a3 = a2 + TB; const char* b3 = b2 + TB;
        RESCALE(t >> 1);
        LDB(B0, 0, 0); SCHED; LDA(At, 0, 0); STAGE(SA(1, 1), a1 + HTB);
        WAIT_L(8); BAR; WAIT_L(0); MMA(0, 0, At, B0); BAR; SCHED;
        LDB(B1, 0, 1); STAGE(SB(0, 0), b2);
        BAR; WAIT_L(0); MMA(0, 1, At, B1); BAR;
        LDA(At, 0, 1); STAGE(SA(0, 0), a2);
        BAR; WAIT_L(0); MMA(1, 0, At, B0); BAR; SCHED;
        STAGE(SB(0, 1), b2 + HTB);
        WAIT_V(6); BAR; MMA(1, 1, At, B1); BAR;
        LDB(B0, 1, 0); SCHED; LDA(At, 1, 0); STAGE(SA(0, 1), a2 + HTB);
        WAIT_L(8); BAR; WAIT_L(0); MMA(0, 0, At, B0); BAR; SCHED;
        LDB(B1, 1, 1); STAGE(SB(1, 0), b3);
        BAR; WAIT_L(0); MMA(0, 1, At, B1); BAR;
        LDA(At, 1, 1); STAGE(SA(1, 0), a3);
        BAR; WAIT_L(0); MMA(1, 0, At, B0); BAR; SCHED;
        STAGE(SB(1, 1), b3 + HTB);
        WAIT_V(6); BAR; MMA(1, 1, At, B1); BAR;
    }
    WAIT_V(0);
    if (wr == 0) BAR;
    BAR;
    STAMP(4);
    int tz = threadIdx.x; asm volatile("" : "+v"(tz));
    const int lane2 = tz & 63, fr2 = lane2 & 15, fq2 = lane2 >> 4;
    const int owner = mt >> 1;
    float* obase = p.out[owner] + (size_t)((mt & 1) * BM) * NG + (size_t)p.rank * NS + nt * BM;
    const float qnan = __builtin_nanf("");
    __amdgpu_buffer_rsrc_t ors = make_rsrc(obase);
#pragma unroll
    for (int ai = 0; ai < 2; ++ai) {
#pragma unroll
        for (int m = 0; m < 4; ++m) {
            const int r = wr * 64 + m * 16 + fr2;
#pragma unroll
            for (int bj = 0; bj < 2; ++bj)
#pragma unroll
                for (int n = 0; n < 2; ++n) {
                    f32x4 a = acc[ai][bj][m][n], o;
#pragma unroll
                    for (int j = 0; j < 4; ++j) o[j] = silu_f(ldexpf(a[j], -ecur));
                    const int ch = bj * 32 + wc * 8 + n * 4 + fq2;
                    *(LAS f32x4*)(lds + r * 1024 + ((ch ^ (r & 7)) << 4)) = o;
                }
        }
        if (ai == 0 && tz == 0) {
            wait_flag(ctl(myws, L_PDONE), call * (unsigned)NPROD, ctl(myws, L_TMO), call);
            *(LAS unsigned*)(lds + LDS_BYTES) = __hip_atomic_load(ctl(myws, L_TMO), __ATOMIC_RELAXED, __HIP_MEMORY_SCOPE_AGENT) == call ? 1u : 0u;
        }
        __syncthreads();
        const bool bad = *(LAS unsigned*)(lds + LDS_BYTES) != 0u;
#pragma unroll 4
        for (int i = 0; i < 16; ++i) {
            const int r = wid * 16 + i;
            f32x4 o = *(LAS f32x4*)(lds + r * 1024 + ((lane2 ^ (r & 7)) << 4));
            if (bad) o = (f32x4){qnan, qnan, qnan, qnan};
            st16_wt(ors, (unsigned)(((ai * HALF + r) * NG + lane2 * 4) * 4), o);
            asm volatile("s_waitcnt vmcnt(0)" ::: "memory");
        }
        __syncthreads();
    }
    STAMP(5);
    asm volatile("s_waitcnt vmcnt(0)" ::: "memory");
    __syncthreads();
    if (threadIdx.x == 0) sys_add(ctl(p.ws[owner], L_DONE), 1u);
    STAMP(6);
#undef SA
#undef SB
#undef STAGE
#undef LDA
#undef LDB
#undef MMA
#undef RESCALE
}

}

__global__ void __launch_bounds__(NTHR, 2) fused_kernel(int phase, Params p) {
    extern __shared__ __attribute__((aligned(16))) unsigned char smem[];
    LAS unsigned char* lds = (LAS unsigned char*)smem;
    char* myws = p.ws[p.rank];
    unsigned* tmo = ctl(myws, L_TMO);
    if (phase == 0) {
        const int b = blockIdx.x;
        f32x4 va[16], vb[16];
        UnitId UA = unit_id(p, b), UB = UA;
        load_unit(p, UA, va);
        const unsigned call = call_number(myws, L_TICK0, gridDim.x, (LAS unsigned*)(lds + LDS_BYTES - 128));
        const int par = call & 1;
        if (threadIdx.x == 0) { __builtin_amdgcn_fence(__ATOMIC_ACQUIRE, ""); asm volatile("s_waitcnt vmcnt(0)" ::: "memory"); }
        __syncthreads();
        STAMP(0);
        const int pidx = p.prod_of_block[b] - 1;
        {
            int unext = pidx >= 0 ? GRID + pidx : 1024;
            unsigned* fprev = nullptr;
#define PSTEP(UC, vc, UN, vn) { \
                const bool more = unext < 1024; \
                if (more) { UN = unit_id(p, unext); load_unit(p, UN, vn); } \
                float am = 0.f; \
                _Pragma("unroll") for (int q = 0; q < 16; ++q) am = umaxf(am, abs4max(vc[q])); \
                am = wg_amax(am, (LAS float*)(lds + LDS_BYTES - 64)); \
                store_unit(p, par, UC, vc, scale_exp(am), lds); \
                if (fprev) { \
                    if (more) asm volatile("s_waitcnt vmcnt(24)" ::: "memory"); else asm volatile("s_waitcnt vmcnt(8)" ::: "memory"); \
                    if ((threadIdx.x & 63) == 0) sys_add(fprev, 1u); \
                } \
                fprev = ctl(p.ws[UC.dst], UC.isx ? L_XRDY + (UC.j & 3) * 128 : L_WRDY + wset(UC.j) * 128); \
                unext += NPROD; \
                if (!more) break; }
            while (true) {
                PSTEP(UA, va, UB, vb)
                PSTEP(UB, vb, UA, va)
            }
#undef PSTEP
            asm volatile("s_waitcnt vmcnt(0)" ::: "memory");
            if ((threadIdx.x & 63) == 0) sys_add(fprev, 1u);
            if (pidx >= 0 && threadIdx.x == 0) __hip_atomic_fetch_add(ctl(myws, L_PDONE), 1u, __ATOMIC_RELAXED, __HIP_MEMORY_SCOPE_AGENT);
            __syncthreads();
            STAMP(1);
        }
        const int tl = p.tile_of_block[b], mt = tl >> 4, nt = tl & 15;
        STAMP(2);
        if (threadIdx.x == 0) {
            wait_flag(ctl(myws, L_XRDY + (mt & 3) * 128), call * 1024u, tmo, call);
            wait_flag(ctl(myws, L_WRDY + wset(nt) * 128), call * (nt < 12 ? 768u : 512u), tmo, call);
            __builtin_amdgcn_fence(__ATOMIC_ACQUIRE, "agent");
            asm volatile("s_waitcnt vmcnt(0)" ::: "memory");
        }
        __syncthreads();
        STAMP(3);
        gemm_tile(p, par, mt, nt, call, lds);
        if (threadIdx.x == 0) {
            const unsigned t = __hip_atomic_fetch_add(ctl(myws, L_FIN), 1u, __ATOMIC_RELAXED, __HIP_MEMORY_SCOPE_AGENT);
            if (t + 1 == call * (unsigned)GRID) {
                STAMP(7);
                wait_flag(ctl(myws, L_DONE), call * (unsigned)(NR * 2 * NNT), tmo, call);
                __builtin_amdgcn_fence(__ATOMIC_ACQUIRE, "");
                if (__hip_atomic_load(tmo, __ATOMIC_RELAXED, __HIP_MEMORY_SCOPE_AGENT) == call)
                    p.out[p.rank][(size_t)p.rank * NS] = __builtin_nanf("");
                STAMP(8);
            }
        }
    }
}

extern "C" void kernel_launch(int rank, int nranks, ncclComm_t comm,
                              hipStream_t stream,
                              void* const* d_in, const int* in_sizes, int n_in,
                              void* d_out, int out_size,
                              const int* in_dtypes, int out_dtype,
                              void* const* d_out_all,
                              void* const* d_ws_all, size_t ws_bytes) {
    if (nranks != NR || ws_bytes < WS_NEED || n_in < 2) { fprintf(stderr, "unexpected configuration\n"); return; }
    static bool attr_set = false;
    if (!attr_set) {
        HIP_CHECK(hipFuncSetAttribute((const void*)fused_kernel, hipFuncAttributeMaxDynamicSharedMemorySize, LDS_ALLOC));
        attr_set = true;
    }
    Params p;
    p.x = (const float*)d_in[0];
    p.w = (const float*)d_in[1];
    for (int i = 0; i < NR; ++i) { p.out[i] = (float*)d_out_all[i]; p.ws[i] = (char*)d_ws_all[i]; }
    p.rank = rank;
    static const int q1[8] = {0, 2, 0, 2, 1, 1, 3, 3}, T1[8] = {0, 0, 1, 1, 0, 1, 0, 1};
    for (int b = 0; b < GRID; ++b) {
        const int x = b & 7, lb = b >> 3;
        int mt, nt;
        if (lb < 8) { mt = (x & 3) + 4 * (lb & 3); nt = 12 + 2 * (x >> 2) + (lb >> 2); }
        else if (lb < 20) { const int e = lb - 8; mt = q1[x] + 4 * (e & 3); nt = 3 * T1[x] + (e >> 2); }
        else { const int e = lb - 20; mt = (x & 3) + 4 * (e & 3); nt = 3 * (2 + (x >> 2)) + (e >> 2); }
        p.tile_of_block[b] = (unsigned char)((mt << 4) | nt);
        p.prod_of_block[b] = lb < 8 ? (unsigned char)(1 + lb * 8 + x) : 0;
    }
    fused_kernel<<<GRID, NTHR, LDS_ALLOC, stream>>>(0, p);
    HIP_CHECK(hipGetLastError());
}
```
